```python
import numpy as np

kernel_name = "gpu_gemm_rs_m4096_k4096_n4096_e5m2e5m2_dqsilu"


ABSMAX_RTOL = 0.05

COMPUTE_DTYPE = "float32"
EPILOGUE = "dequant_silu"
IN_QUANT = None
OUT_QUANT = None
IN_FP8 = "e5m2e5m2"
IN_DTYPES = {"x": "e5m2", "w_mat": "e5m2",
             "scale_x": "float32", "scale_w": "float32"}
WORLD_SIZE = 8

def _snap_fp8(x, mant_bits, max_abs, min_normal):
    a = np.abs(x.astype(np.float64))
    _m, _e = np.frexp(a)
    q = np.where(a >= min_normal, np.ldexp(1.0, _e - 1 - mant_bits),
                 min_normal * 2.0 ** -mant_bits)
    snapped = np.minimum(np.round(a / q) * q, max_abs)
    return (np.sign(x) * snapped).astype(np.float32)

def setup_inputs():

    x = _snap_fp8(np.random.randn(4096, 4096).astype(np.float32), 2, 57344.0, 2.0 ** -14)
    w_mat = _snap_fp8(np.random.randn(4096, 4096).astype(np.float32), 2, 57344.0, 2.0 ** -14)
    scale_x = (0.02 * (1.0 + np.random.rand(1))).astype(np.float32)
    scale_w = (0.02 * (1.0 + np.random.rand(1))).astype(np.float32)
    return {"x": x, "w_mat": w_mat, "scale_x": scale_x, "scale_w": scale_w}

def reference(x, w_mat, scale_x, scale_w):
    acc = (x @ w_mat).astype(np.float32)
    y = (acc * (scale_x * scale_w)).astype(np.float32)
    return (y / (1.0 + np.exp(-np.clip(y, -60.0, 60.0)))).astype(np.float32)
```

```cpp
#include <hip/hip_runtime.h>
#include <rccl/rccl.h>
#include <cstdio>
#include <cstdint>

typedef unsigned u32x4 __attribute__((ext_vector_type(4)));
typedef int i32x8 __attribute__((ext_vector_type(8)));
typedef int i32x4 __attribute__((ext_vector_type(4)));
typedef float f32x4 __attribute__((ext_vector_type(4)));

#define WIN 5
#define RING 16

constexpr int NR = 8;
constexpr int NWG = 256;
constexpr int NTHREADS = 384;
constexpr size_t XBUF = 2u << 20, X_OFF = 0;
constexpr size_t WBUF = 2u << 20, W_OFF = 4u << 20;
constexpr size_t C_OFF = 8u << 20;
constexpr size_t CTR_OFF = C_OFF;
constexpr size_t WFLAG_OFF = C_OFF + (64u << 10);
constexpr size_t XFLAG_OFF = C_OFF + (96u << 10);
constexpr size_t TMO_OFF = C_OFF + (100u << 10);
constexpr size_t INV_OFF = C_OFF + (104u << 10);
constexpr size_t WS_NEED = C_OFF + (1u << 20);
constexpr unsigned long long DEADLINE_TICKS = 50000000ull;

#define STAMP(i) do { } while (0)

struct Params {
    const uint8_t* x;
    const uint8_t* w;
    const float* sx;
    const float* sw;
    float* out;
    char* ws[NR];
    int rank;
};

#define RLX_SYS __ATOMIC_RELAXED, __HIP_MEMORY_SCOPE_SYSTEM
#define RLX_AGT __ATOMIC_RELAXED, __HIP_MEMORY_SCOPE_AGENT
#define RLX_WG __ATOMIC_RELAXED, __HIP_MEMORY_SCOPE_WORKGROUP
#define CBAR() asm volatile("" ::: "memory")

enum { L_READY = 0, L_CONS = 1, L_XOWN = 5, L_XALL = 6, L_ABORT = 7, L_WORDS = 16 };

__device__ __forceinline__ void glds_piece(unsigned voff, const char* sbase, unsigned ldsdst) {
    unsigned keep;
    asm volatile("s_mov_b32 %0, m0\n\ts_mov_b32 m0, %2\n\ts_nop 0\n\tglobal_load_lds_dwordx4 %1, %3\n\ts_mov_b32 m0, %0"
                 : "=&s"(keep) : "v"(voff), "s"(ldsdst), "s"(sbase) : "memory");
}
__device__ __forceinline__ void lds_store_asm(unsigned addr, unsigned val) {
    asm volatile("ds_write_b32 %0, %1" :: "v"(addr), "v"(val) : "memory");
}
__device__ __forceinline__ unsigned lds_load_asm(unsigned addr) {
    unsigned v;
    asm volatile("ds_read_b32 %0, %1\n\ts_waitcnt lgkmcnt(0)" : "=&v"(v) : "v"(addr) : "memory");
    return v;
}

__device__ __forceinline__ bool wait_flags(const unsigned* f, bool active, unsigned call, unsigned* tmo) {
    const unsigned long long t0 = __builtin_amdgcn_s_memrealtime();
    for (unsigned it = 1;; ++it) {
        unsigned v = active ? __hip_atomic_load(f, RLX_SYS) : call;
        if (__all(v >= call)) return true;
        if ((it & 63u) == 0u) {
            if (__hip_atomic_load(tmo, RLX_AGT) == call || __builtin_amdgcn_s_memrealtime() - t0 > DEADLINE_TICKS) {
                __hip_atomic_store(tmo, call, RLX_AGT);
                return false;
            }
        }
        __builtin_amdgcn_s_sleep(1);
    }
}
__device__ __forceinline__ void wait_lds(unsigned addr, unsigned need, unsigned abort_addr) {
    if (__builtin_amdgcn_readfirstlane(lds_load_asm(addr)) >= need) return;
    const unsigned long long t0 = __builtin_amdgcn_s_memrealtime();
    for (;;) {
        __builtin_amdgcn_s_sleep(1);
        if (__builtin_amdgcn_readfirstlane(lds_load_asm(addr)) >= need) return;
        if (__builtin_amdgcn_s_memrealtime() - t0 > 2 * DEADLINE_TICKS) {
            lds_store_asm(abort_addr, 1u);
            return;
        }
    }
}

__global__ void __launch_bounds__(NTHREADS) fused_gemm_silu(Params p) {
    __shared__ __attribute__((aligned(16))) char lds[RING * 8192 + L_WORDS * 4];
    const int tid = threadIdx.x;
    const int lane = tid & 63;
    const int wave = __builtin_amdgcn_readfirstlane(tid >> 6);
    const int b = blockIdx.x;
    const int rank = p.rank;
    char* me = p.ws[rank];
    const unsigned lds0 = (unsigned)(size_t)lds;
    const unsigned ctl = lds0 + RING * 8192;
    unsigned* tmo = (unsigned*)(me + TMO_OFF);

    const int idx = b >> 3;
    const int G = (b & 7) * 8 + (idx >> 2);
    const int rg = idx & 3;

    if (wave == 0) STAMP(0);
    if (tid < L_WORDS) ((unsigned*)(lds + RING * 8192))[tid] = 0u;
    unsigned* ctr = (unsigned*)(me + CTR_OFF) + b * 8 + wave;
    const unsigned cprev = __hip_atomic_load(ctr, RLX_AGT);

    if (wave == 0) {
        const int Ge = b >> 2, se = b & 3;
        const int i = lane & 15, g = lane >> 4;
        const uint8_t* src = p.w + (size_t)(se * 128 + g * 16) * 4096 + Ge * 64 + i * 4;
        unsigned v[2][16];
#pragma unroll
        for (int h = 0; h < 2; ++h)
#pragma unroll
            for (int r = 0; r < 16; ++r) v[h][r] = *(const unsigned*)(src + (size_t)(h * 64 + r) * 4096);
        __syncthreads();
        const unsigned call = __builtin_amdgcn_readfirstlane(cprev) + 1u;
        if (lane == 0) __hip_atomic_store(ctr, call, RLX_AGT);
        __amdgpu_buffer_rsrc_t rd = __builtin_amdgcn_make_buffer_rsrc(me + W_OFF + (call & 1u) * WBUF, 0, (unsigned)WBUF, 0x00020000);
        const unsigned obase = (unsigned)(Ge * 4 + se) * 8192u + lane * 16u;
#pragma unroll
        for (int h = 0; h < 2; ++h) {
            u32x4 o[4];
#pragma unroll
            for (int d = 0; d < 4; ++d) {
                const unsigned r0 = v[h][4 * d], r1 = v[h][4 * d + 1], r2 = v[h][4 * d + 2], r3 = v[h][4 * d + 3];
                const unsigned t0 = __builtin_amdgcn_perm(r1, r0, 0x05010400u), t1 = __builtin_amdgcn_perm(r1, r0, 0x07030602u);
                const unsigned t2 = __builtin_amdgcn_perm(r3, r2, 0x05010400u), t3 = __builtin_amdgcn_perm(r3, r2, 0x07030602u);
                o[0][d] = __builtin_amdgcn_perm(t2, t0, 0x05040100u);
                o[1][d] = __builtin_amdgcn_perm(t2, t0, 0x07060302u);
                o[2][d] = __builtin_amdgcn_perm(t3, t1, 0x05040100u);
                o[3][d] = __builtin_amdgcn_perm(t3, t1, 0x07060302u);
            }
#pragma unroll
            for (int q = 0; q < 4; ++q)
                __builtin_amdgcn_raw_buffer_store_b128(o[q], rd, obase + (unsigned)(q * 2 + h) * 1024u, 0, 17);
        }
        asm volatile("s_waitcnt vmcnt(0)" ::: "memory");
        if (lane < NR)
            __hip_atomic_store((unsigned*)(p.ws[lane] + WFLAG_OFF) + Ge * 32 + rank * 4 + se, call, RLX_SYS);
        STAMP(1);
    } else if (wave == 1) {
        const int dest = (b + (b >> 3)) & 7;
        const int xr = idx & 3, xs = (idx >> 2) & 3, xh = idx >> 4;
        const uint8_t* src = p.x + (size_t)(dest * 512 + xr * 128 + xh * 64 + (lane & 15)) * 512 + xs * 128 + (lane >> 4) * 16;
        u32x4 v[8];
#pragma unroll
        for (int t = 0; t < 4; ++t)
#pragma unroll
            for (int h = 0; h < 2; ++h) v[t * 2 + h] = *(const u32x4*)(src + (size_t)t * 16 * 512 + h * 64);
        __syncthreads();
        const unsigned call = __builtin_amdgcn_readfirstlane(cprev) + 1u;
        if (lane == 0) __hip_atomic_store(ctr, call, RLX_AGT);
        char* dws = p.ws[dest];
        __amdgpu_buffer_rsrc_t rd = __builtin_amdgcn_make_buffer_rsrc(dws + X_OFF + (call & 1u) * XBUF, 0, (unsigned)XBUF, 0x00020000);
        const unsigned obase = (unsigned)rank * 262144u + (unsigned)xs * 65536u + (unsigned)(xr * 8 + xh * 4) * 2048u + lane * 16u;
#pragma unroll
        for (int u = 0; u < 8; ++u) __builtin_amdgcn_raw_buffer_store_b128(v[u], rd, obase + (unsigned)u * 1024u, 0, 17);
        asm volatile("s_waitcnt vmcnt(0)" ::: "memory");
        if (lane == 0)
            __hip_atomic_store((unsigned*)(dws + XFLAG_OFF) + xr * 64 + (rank * 4 + xs) * 2 + xh, call, RLX_SYS);
        STAMP(2);
    } else {
        __syncthreads();
    }
    const unsigned call = __builtin_amdgcn_readfirstlane(cprev) + 1u;
    if (wave >= 2 && lane == 0) __hip_atomic_store(ctr, call, RLX_AGT);
    const unsigned buf = call & 1u;

    if (wave == 4) {
        const unsigned voff = lane * 16u;
        const unsigned* wf = (const unsigned*)(me + WFLAG_OFF) + G * 32 + (lane & 31);
        const size_t goff = W_OFF + (size_t)buf * WBUF + (size_t)G * 32768;
        bool ok = wait_flags(wf, lane < 32 && (lane >> 2) == rank, call, tmo);
        STAMP(3);
        if (ok) {
            const char* sb = me + goff;
#pragma unroll
            for (int u = 0; u < 32; ++u) glds_piece(voff, sb + u * 1024, lds0 + u * 1024);
            asm volatile("s_waitcnt vmcnt(0)" ::: "memory");
            lds_store_asm(ctl + L_READY * 4, 4u);
            STAMP(4);
            ok = wait_flags(wf, lane < 32, call, tmo);
        }
        STAMP(5);
        if (ok) {
            const unsigned xcc = __builtin_amdgcn_s_getreg(20 | (3 << 11)) & 15u;
            const unsigned inv = __hip_atomic_load((const unsigned*)(me + INV_OFF) + xcc * 32, RLX_AGT);
            if (__builtin_amdgcn_readfirstlane(inv) != call) __builtin_amdgcn_fence(__ATOMIC_ACQUIRE, "");
            asm volatile("s_waitcnt vmcnt(0)" ::: "memory");
            STAMP(6);
            int i7 = G % 7, s = 0;
            for (int J = 4; J < 32; ++J) {
                const int src = (rank + 1 + i7) & 7;
                const char* sb = p.ws[src] + goff + s * 8192;
                const unsigned dst = lds0 + (unsigned)(J % RING) * 8192u;
                if (J >= RING) {
#pragma unroll
                    for (int w = 0; w < 4; ++w) wait_lds(ctl + (L_CONS + w) * 4, (unsigned)(J - RING + 1), ctl + L_ABORT * 4);
                }
#pragma unroll
                for (int u = 0; u < 8; ++u) {
                    glds_piece(voff, sb + u * 1024, dst + u * 1024);
                    asm volatile("s_waitcnt vmcnt(%0)" :: "n"(WIN - 1) : "memory");
                    if (WIN == 1 ? (u == 7) : (u == WIN - 2)) lds_store_asm(ctl + L_READY * 4, (unsigned)(WIN == 1 ? J + 1 : J));
                }
                if (++i7 == 7) i7 = 0;
                if (i7 == G % 7) ++s;
            }
            asm volatile("s_waitcnt vmcnt(0)" ::: "memory");
        }
        if (!ok) lds_store_asm(ctl + L_ABORT * 4, 1u);
        lds_store_asm(ctl + L_READY * 4, 32u);
        STAMP(7);
        return;
    }
    if (wave == 5) {
        const unsigned* xf = (const unsigned*)(me + XFLAG_OFF) + rg * 64 + lane;
        if (b < 8) {
            __builtin_amdgcn_fence(__ATOMIC_ACQUIRE, "");
            asm volatile("s_waitcnt vmcnt(0)" ::: "memory");
            const unsigned xcc = __builtin_amdgcn_s_getreg(20 | (3 << 11)) & 15u;
            if (lane == 0) __hip_atomic_store((unsigned*)(me + INV_OFF) + xcc * 32, call, RLX_AGT);
        }
        bool ok = wait_flags(xf, (lane >> 3) == rank, call, tmo);
        if (!ok) lds_store_asm(ctl + L_ABORT * 4, 1u);
        lds_store_asm(ctl + L_XOWN * 4, 1u);
        STAMP(8);
        if (ok) ok = wait_flags(xf, true, call, tmo);
        if (!ok) lds_store_asm(ctl + L_ABORT * 4, 1u);
        lds_store_asm(ctl + L_XALL * 4, 1u);
        STAMP(9);
        return;
    }

    const float scale = p.sx[0] * p.sw[0];
    f32x4 acc[2][4];
#pragma unroll
    for (int t = 0; t < 2; ++t)
#pragma unroll
        for (int q = 0; q < 4; ++q) acc[t][q] = (f32x4){0.f, 0.f, 0.f, 0.f};

    const char* xb = me + X_OFF + (size_t)buf * XBUF + (size_t)(rg * 8 + wave * 2) * 2048 + lane * 16;
    i32x4 A[4][4];
    auto loadA = [&](int slot, int src, int s) {
        const char* a = xb + (size_t)src * 262144 + (size_t)s * 65536;
#pragma unroll
        for (int u = 0; u < 4; ++u) A[slot][u] = *(const i32x4*)(a + u * 1024);
    };
    auto step = [&](int slot, int J) {
        wait_lds(ctl + L_READY * 4, (unsigned)(J + 1), ctl + L_ABORT * 4);
        CBAR();
        const char* bs = lds + (J % RING) * 8192 + lane * 16;
        i32x8 B[4];
#pragma unroll
        for (int q = 0; q < 4; ++q) {
            const i32x4 lo = *(const i32x4*)(bs + (q * 2) * 1024), hi = *(const i32x4*)(bs + (q * 2 + 1) * 1024);
            B[q] = __builtin_shufflevector(lo, hi, 0, 1, 2, 3, 4, 5, 6, 7);
        }
#pragma unroll
        for (int t = 0; t < 2; ++t) {
            const i32x8 a = __builtin_shufflevector(A[slot][t * 2], A[slot][t * 2 + 1], 0, 1, 2, 3, 4, 5, 6, 7);
#pragma unroll
            for (int q = 0; q < 4; ++q)
                acc[t][q] = __builtin_amdgcn_mfma_scale_f32_16x16x128_f8f6f4(a, B[q], acc[t][q], 1, 1, 0, 0x7f7f7f7f, 0, 0x7f7f7f7f);
        }
        asm volatile("" : "+v"(acc[0][0]), "+v"(acc[0][1]), "+v"(acc[0][2]), "+v"(acc[0][3]),
                          "+v"(acc[1][0]), "+v"(acc[1][1]), "+v"(acc[1][2]), "+v"(acc[1][3]) :: "memory");
        lds_store_asm(ctl + (L_CONS + wave) * 4, (unsigned)(J + 1));
    };

    wait_lds(ctl + L_XOWN * 4, 1u, ctl + L_ABORT * 4);
    CBAR();
#pragma unroll
    for (int u = 0; u < 4; ++u) loadA(u, rank, u);
#pragma unroll
    for (int u = 0; u < 4; ++u) step(u, u);
    if (wave == 2) STAMP(10);
    wait_lds(ctl + L_XALL * 4, 1u, ctl + L_ABORT * 4);
    CBAR();
    if (wave == 2) STAMP(11);
    const int g7 = G % 7;
    int pi = g7, ps = 0;
    auto adv = [&]() { if (++pi == 7) pi = 0; if (pi == g7) ++ps; };
#pragma unroll
    for (int u = 0; u < 4; ++u) { loadA(u, (rank + 1 + pi) & 7, ps); adv(); }
    for (int J0 = 4; J0 < 32; J0 += 4) {
#pragma unroll
        for (int u = 0; u < 4; ++u) {
            step(u, J0 + u);
            loadA(u, (rank + 1 + pi) & 7, ps & 3);
            adv();
        }
    }
    if (wave == 2) STAMP(12);

    const bool bad = __builtin_amdgcn_readfirstlane(lds_load_asm(ctl + L_ABORT * 4)) != 0u;
    float* o = p.out + (size_t)(rg * 128 + wave * 32 + (lane >> 4) * 4) * 4096 + G * 64 + (lane & 15) * 4;
#pragma unroll
    for (int t = 0; t < 2; ++t)
#pragma unroll
        for (int r = 0; r < 4; ++r) {
            f32x4 y;
#pragma unroll
            for (int q = 0; q < 4; ++q) {
                const float v = acc[t][q][r] * scale;
                const float c = fminf(fmaxf(v, -60.f), 60.f);
                y[q] = bad ? __builtin_nanf("") : v * __builtin_amdgcn_rcpf(1.f + __expf(-c));
            }
            *(f32x4*)(o + (size_t)(t * 16 + r) * 4096) = y;
        }
}

extern "C" void kernel_launch(int rank, int nranks, ncclComm_t comm, hipStream_t stream,
                              void* const* d_in, const int* in_sizes, int n_in,
                              void* d_out, int out_size,
                              const int* in_dtypes, int out_dtype,
                              void* const* d_out_all,
                              void* const* d_ws_all, size_t ws_bytes) {
    if (nranks != NR || (unsigned)rank >= (unsigned)NR || ws_bytes < WS_NEED || n_in < 4 ||
        in_sizes[0] != 4096 * 512 || in_sizes[1] != 512 * 4096 || out_size != 512 * 4096) {
        fprintf(stderr, "kernel_launch: unexpected problem geometry\n");
        return;
    }
    Params p;
    p.x = (const uint8_t*)d_in[0];
    p.w = (const uint8_t*)d_in[1];
    p.sx = (const float*)d_in[2];
    p.sw = (const float*)d_in[3];
    p.out = (float*)d_out;
    for (int r = 0; r < NR; ++r) p.ws[r] = (char*)d_ws_all[r];
    p.rank = rank;
    hipLaunchKernelGGL(fused_gemm_silu, dim3(NWG), dim3(NTHREADS), 0, stream, p);
}
```

```cpp
#include <hip/hip_runtime.h>
#include <rccl/rccl.h>
#include <cstdio>
#include <cstdint>

typedef unsigned u32x4 __attribute__((ext_vector_type(4)));
typedef int i32x8 __attribute__((ext_vector_type(8)));
typedef int i32x4 __attribute__((ext_vector_type(4)));
typedef float f32x4 __attribute__((ext_vector_type(4)));

#define RING 16
#define WIN 8
constexpr int NR = 8;
constexpr int NWG = 256;
constexpr int NTHREADS = 384;
constexpr size_t XBUF = 2u << 20, X_OFF = 0;
constexpr size_t WBUF = 2u << 20, W_OFF = 4u << 20;
constexpr size_t C_OFF = 8u << 20;
constexpr size_t CTR_OFF = C_OFF;
constexpr size_t WFLAG_OFF = C_OFF + (64u << 10);
constexpr size_t XFLAG_OFF = C_OFF + (96u << 10);
constexpr size_t TMO_OFF = C_OFF + (100u << 10);
constexpr size_t INV_OFF = C_OFF + (104u << 10);
constexpr size_t WS_NEED = C_OFF + (1u << 20);
constexpr unsigned long long DEADLINE_TICKS = 50000000ull;

#define STAMP(i) do { } while (0)

struct Params {
    const uint8_t* x; const uint8_t* w; const float* sx; const float* sw; float* out;
    char* ws[NR];
    int rank;
};
#define RLX_SYS __ATOMIC_RELAXED, __HIP_MEMORY_SCOPE_SYSTEM
#define RLX_AGT __ATOMIC_RELAXED, __HIP_MEMORY_SCOPE_AGENT
#define CBAR() asm volatile("" ::: "memory")
enum { L_READY = 0, L_CONS = 1, L_XOWN = 5, L_XALL = 6, L_ABORT = 7, L_WORDS = 16 };

__device__ __forceinline__ void glds_piece(unsigned voff, const char* sbase, unsigned ldsdst) {
    unsigned keep;
    asm volatile("s_mov_b32 %0, m0\n\ts_mov_b32 m0, %2\n\ts_nop 0\n\tglobal_load_lds_dwordx4 %1, %3\n\ts_mov_b32 m0, %0"
                 : "=&s"(keep) : "v"(voff), "s"(ldsdst), "s"(sbase) : "memory");
}
__device__ __forceinline__ void lds_store_asm(unsigned addr, unsigned val) { asm volatile("ds_write_b32 %0, %1" :: "v"(addr), "v"(val) : "memory"); }
__device__ __forceinline__ unsigned lds_load_asm(unsigned addr) {
    unsigned v;
    asm volatile("ds_read_b32 %0, %1\n\ts_waitcnt lgkmcnt(0)" : "=&v"(v) : "v"(addr) : "memory");
    return v;
}
__device__ __forceinline__ bool wait_flags(const unsigned* f, bool active, unsigned call, unsigned* tmo) {
    const unsigned long long t0 = __builtin_amdgcn_s_memrealtime();
    for (unsigned it = 1;; ++it) {
        unsigned v = active ? __hip_atomic_load(f, RLX_SYS) : call;
        if (__all(v >= call)) return true;
        if ((it & 63u) == 0u) {
            if (__hip_atomic_load(tmo, RLX_AGT) == call || __builtin_amdgcn_s_memrealtime() - t0 > DEADLINE_TICKS) {
                __hip_atomic_store(tmo, call, RLX_AGT);
                return false;
            }
        }
        __builtin_amdgcn_s_sleep(1);
    }
}
__device__ __forceinline__ void wait_lds(unsigned addr, unsigned need, unsigned abort_addr) {
    if (__builtin_amdgcn_readfirstlane(lds_load_asm(addr)) >= need) return;
    const unsigned long long t0 = __builtin_amdgcn_s_memrealtime();
    for (;;) {
        __builtin_amdgcn_s_sleep(1);
        if (__builtin_amdgcn_readfirstlane(lds_load_asm(addr)) >= need) return;
        if (__builtin_amdgcn_s_memrealtime() - t0 > 2 * DEADLINE_TICKS) { lds_store_asm(abort_addr, 1u); return; }
    }
}

__device__ __forceinline__ void pull_loop(const Params& p, int rank, int G, size_t goff, unsigned lds0, unsigned ctl, unsigned voff) {
    int i7 = G % 7, kc = 0, cnt = 0;
    for (int ss = 0; ss < 7; ++ss) {
        const char* cb[8];
#pragma unroll
        for (int c = 0; c < 8; ++c) {
            cb[c] = p.ws[(rank + 1 + i7) & 7] + goff + (kc >> 1) * 8192 + (kc & 1) * 1024;
            if (++i7 == 7) i7 = 0;
            if (++cnt == 7) { cnt = 0; ++kc; }
        }
#pragma unroll
        for (int q = 0; q < 4; ++q) {
            const int J = 4 + ss * 4 + q;
            const unsigned dst = lds0 + (unsigned)(J % RING) * 8192u;
            if (J >= RING) {
#pragma unroll
                for (int w = 0; w < 4; ++w) wait_lds(ctl + (L_CONS + w) * 4, (unsigned)(J - RING + 1), ctl + L_ABORT * 4);
            }
#pragma unroll
            for (int c = 0; c < 8; ++c) {
                glds_piece(voff, cb[c] + q * 2048, dst + c * 1024);
                asm volatile("s_waitcnt vmcnt(%0)" :: "n"(WIN - 1) : "memory");
                if (c == WIN - 2) lds_store_asm(ctl + L_READY * 4, (unsigned)J);
            }
        }
    }
}

__global__ void __launch_bounds__(NTHREADS) fused_gemm_silu(Params p) {
    __shared__ __attribute__((aligned(16))) char lds[RING * 8192 + L_WORDS * 4];
    const int tid = threadIdx.x;
    const int lane = tid & 63;
    const int wave = __builtin_amdgcn_readfirstlane(tid >> 6);
    const int b = blockIdx.x;
    const int rank = p.rank;
    char* me = p.ws[rank];
    const unsigned lds0 = (unsigned)(size_t)lds;
    const unsigned ctl = lds0 + RING * 8192;
    unsigned* tmo = (unsigned*)(me + TMO_OFF);
    const int idx = b >> 3;
    const int G = (b & 7) * 8 + (idx >> 2);
    const int rg = idx & 3;

    if (wave == 0) STAMP(0);
    if (tid < L_WORDS) ((unsigned*)(lds + RING * 8192))[tid] = (tid == L_READY) ? 4u : 0u;
    unsigned* ctr = (unsigned*)(me + CTR_OFF) + b * 8 + wave;
    const unsigned cprev = __hip_atomic_load(ctr, RLX_AGT);

    if (wave == 0) {
        const int Ge = b >> 2, se = b & 3;
        const int i = lane & 15, g = lane >> 4;
        const uint8_t* src = p.w + (size_t)(se * 128 + g * 16) * 4096 + Ge * 64 + i * 4;
        unsigned v[2][16];
#pragma unroll
        for (int h = 0; h < 2; ++h)
#pragma unroll
            for (int r = 0; r < 16; ++r) v[h][r] = *(const unsigned*)(src + (size_t)(h * 64 + r) * 4096);
        __syncthreads();
        const unsigned call = __builtin_amdgcn_readfirstlane(cprev) + 1u;
        if (lane == 0) __hip_atomic_store(ctr, call, RLX_AGT);
        __amdgpu_buffer_rsrc_t rd = __builtin_amdgcn_make_buffer_rsrc(me + W_OFF + (call & 1u) * WBUF, 0, (unsigned)WBUF, 0x00020000);
        const unsigned obase = (unsigned)(Ge * 4 + se) * 8192u + lane * 16u;
#pragma unroll
        for (int h = 0; h < 2; ++h) {
            u32x4 o[4];
#pragma unroll
            for (int d = 0; d < 4; ++d) {
                const unsigned r0 = v[h][4 * d], r1 = v[h][4 * d + 1], r2 = v[h][4 * d + 2], r3 = v[h][4 * d + 3];
                const unsigned t0 = __builtin_amdgcn_perm(r1, r0, 0x05010400u), t1 = __builtin_amdgcn_perm(r1, r0, 0x07030602u);
                const unsigned t2 = __builtin_amdgcn_perm(r3, r2, 0x05010400u), t3 = __builtin_amdgcn_perm(r3, r2, 0x07030602u);
                o[0][d] = __builtin_amdgcn_perm(t2, t0, 0x05040100u);
                o[1][d] = __builtin_amdgcn_perm(t2, t0, 0x07060302u);
                o[2][d] = __builtin_amdgcn_perm(t3, t1, 0x05040100u);
                o[3][d] = __builtin_amdgcn_perm(t3, t1, 0x07060302u);
            }
#pragma unroll
            for (int q = 0; q < 4; ++q)
                __builtin_amdgcn_raw_buffer_store_b128(o[q], rd, obase + (unsigned)(q * 2 + h) * 1024u, 0, 17);
        }
        asm volatile("s_waitcnt vmcnt(0)" ::: "memory");
        if (lane < NR)
            __hip_atomic_store((unsigned*)(p.ws[lane] + WFLAG_OFF) + Ge * 32 + rank * 4 + se, call, RLX_SYS);
        STAMP(1);
    } else if (wave == 1) {
        const int dest = (b + idx) & 7;
        const int xr = idx & 3, xs = (idx >> 2) & 3, xh = idx >> 4;
        const uint8_t* src = p.x + (size_t)(dest * 512 + xr * 128 + xh * 64 + (lane & 15)) * 512 + xs * 128 + (lane >> 4) * 16;
        u32x4 v[8];
#pragma unroll
        for (int t = 0; t < 4; ++t)
#pragma unroll
            for (int h = 0; h < 2; ++h) v[t * 2 + h] = *(const u32x4*)(src + (size_t)t * 16 * 512 + h * 64);
        __syncthreads();
        const unsigned call = __builtin_amdgcn_readfirstlane(cprev) + 1u;
        if (lane == 0) __hip_atomic_store(ctr, call, RLX_AGT);
        char* dws = p.ws[dest];
        __amdgpu_buffer_rsrc_t rd = __builtin_amdgcn_make_buffer_rsrc(dws + X_OFF + (call & 1u) * XBUF, 0, (unsigned)XBUF, 0x00020000);
        const unsigned obase = (unsigned)rank * 262144u + (unsigned)xs * 65536u + (unsigned)(xr * 8 + xh * 4) * 2048u + lane * 16u;
#pragma unroll
        for (int u = 0; u < 8; ++u) {
            __builtin_amdgcn_raw_buffer_store_b128(v[u], rd, obase + (unsigned)u * 1024u, 0, 17);
            asm volatile("s_waitcnt vmcnt(0)" ::: "memory");
        }
        asm volatile("s_waitcnt vmcnt(0)" ::: "memory");
        if (lane == 0)
            __hip_atomic_store((unsigned*)(dws + XFLAG_OFF) + xr * 64 + (rank * 4 + xs) * 2 + xh, call, RLX_SYS);
        STAMP(2);
    } else {
        __syncthreads();
    }
    const unsigned call = __builtin_amdgcn_readfirstlane(cprev) + 1u;
    if (wave >= 2 && lane == 0) __hip_atomic_store(ctr, call, RLX_AGT);
    const unsigned buf = call & 1u;

    if (wave == 4) {
        const unsigned voff = lane * 16u;
        const size_t goff = W_OFF + (size_t)buf * WBUF + (size_t)G * 32768;
        const unsigned xcc = __builtin_amdgcn_s_getreg(20 | (3 << 11)) & 15u;
        const unsigned* pf = (const unsigned*)(me + WFLAG_OFF) + G * 32 + (lane & 31);
        bool ok = wait_flags(pf, lane < 32 && (lane >> 2) != rank, call, tmo);
        STAMP(5);
        if (ok) {
            const unsigned inv = __hip_atomic_load((const unsigned*)(me + INV_OFF) + xcc * 32, RLX_AGT);
            if (__builtin_amdgcn_readfirstlane(inv) != call) __builtin_amdgcn_fence(__ATOMIC_ACQUIRE, "");
            asm volatile("s_waitcnt vmcnt(0)" ::: "memory");
            STAMP(6);
            pull_loop(p, rank, G, goff, lds0, ctl, voff);
            asm volatile("s_waitcnt vmcnt(0)" ::: "memory");
        }
        if (!ok) lds_store_asm(ctl + L_ABORT * 4, 1u);
        lds_store_asm(ctl + L_READY * 4, 32u);
        STAMP(7);
        return;
    }
    if (wave == 5) {
        const unsigned* xf = (const unsigned*)(me + XFLAG_OFF) + rg * 64 + lane;
        if (b < 8) {
            __builtin_amdgcn_fence(__ATOMIC_ACQUIRE, "");
            asm volatile("s_waitcnt vmcnt(0)" ::: "memory");
            const unsigned xcc = __builtin_amdgcn_s_getreg(20 | (3 << 11)) & 15u;
            if (lane == 0) __hip_atomic_store((unsigned*)(me + INV_OFF) + xcc * 32, call, RLX_AGT);
        }
        bool ok = wait_flags((const unsigned*)(me + WFLAG_OFF) + G * 32 + rank * 4 + (lane & 3), lane < 4, call, tmo);
        if (ok) {
            const char* sb = me + W_OFF + (size_t)buf * WBUF + (size_t)G * 32768;
            for (int q = 0; q < 4; ++q)
#pragma unroll
                for (int c = 0; c < 8; ++c) glds_piece(lane * 16u, sb + (c >> 1) * 8192 + q * 2048 + (c & 1) * 1024, lds0 + q * 8192 + c * 1024);
            asm volatile("s_waitcnt vmcnt(0)" ::: "memory");
            ok = wait_flags(xf, (lane >> 3) == rank, call, tmo);
        }
        if (!ok) lds_store_asm(ctl + L_ABORT * 4, 1u);
        lds_store_asm(ctl + L_XOWN * 4, 1u);
        STAMP(8);
        if (ok) ok = wait_flags(xf, true, call, tmo);
        if (!ok) lds_store_asm(ctl + L_ABORT * 4, 1u);
        lds_store_asm(ctl + L_XALL * 4, 1u);
        STAMP(9);
        return;
    }

    const float scale = p.sx[0] * p.sw[0];
    f32x4 acc[2][4];
#pragma unroll
    for (int t = 0; t < 2; ++t)
#pragma unroll
        for (int q = 0; q < 4; ++q) acc[t][q] = (f32x4){0.f, 0.f, 0.f, 0.f};
    const char* xb = me + X_OFF + (size_t)buf * XBUF + (size_t)(rg * 8 + wave * 2) * 2048 + lane * 16;
    i32x4 A0[8][2], A1[8][2];
    int pi = G % 7, pk = 0, pc = 0;
    auto loadA = [&](i32x4 (&A)[8][2]) {
#pragma unroll
        for (int c = 0; c < 8; ++c) {
            const char* a = xb + (size_t)((rank + 1 + pi) & 7) * 262144 + (size_t)((pk >> 1) & 3) * 65536 + (pk & 1) * 1024;
            A[c][0] = *(const i32x4*)a;
            A[c][1] = *(const i32x4*)(a + 2048);
            if (++pi == 7) pi = 0;
            if (++pc == 7) { pc = 0; ++pk; }
        }
    };
    auto sstep = [&](i32x4 (&A)[8][2], int J0) {
#pragma unroll
        for (int q = 0; q < 4; ++q) {
            const int J = J0 + q;
            wait_lds(ctl + L_READY * 4, (unsigned)(J + 1), ctl + L_ABORT * 4);
            CBAR();
            const char* bs = lds + (J % RING) * 8192 + lane * 16;
#pragma unroll
            for (int c = 0; c < 8; c += 2) {
                const i32x4 lo = *(const i32x4*)(bs + c * 1024), hi = *(const i32x4*)(bs + (c + 1) * 1024);
                const i32x8 B = __builtin_shufflevector(lo, hi, 0, 1, 2, 3, 4, 5, 6, 7);
#pragma unroll
                for (int t = 0; t < 2; ++t) {
                    const i32x8 a = __builtin_shufflevector(A[c][t], A[c + 1][t], 0, 1, 2, 3, 4, 5, 6, 7);
                    acc[t][q] = __builtin_amdgcn_mfma_scale_f32_16x16x128_f8f6f4(a, B, acc[t][q], 1, 1, 0, 0x7f7f7f7f, 0, 0x7f7f7f7f);
                }
            }
            asm volatile("" : "+v"(acc[0][q]), "+v"(acc[1][q]) :: "memory");
            lds_store_asm(ctl + (L_CONS + wave) * 4, (unsigned)(J + 1));
        }
    };
    wait_lds(ctl + L_XOWN * 4, 1u, ctl + L_ABORT * 4);
    CBAR();
#pragma unroll
    for (int c = 0; c < 8; ++c) {
        const char* a = xb + (size_t)rank * 262144 + (size_t)(c >> 1) * 65536 + (c & 1) * 1024;
        A1[c][0] = *(const i32x4*)a;
        A1[c][1] = *(const i32x4*)(a + 2048);
    }
    sstep(A1, 0);
    wait_lds(ctl + L_XALL * 4, 1u, ctl + L_ABORT * 4);
    CBAR();
    if (wave == 2) STAMP(11);
    loadA(A0);
    for (int J0 = 4; J0 < 28; J0 += 8) {
        loadA(A1);
        sstep(A0, J0);
        loadA(A0);
        sstep(A1, J0 + 4);
    }
    sstep(A0, 28);
    if (wave == 2) STAMP(12);

    const bool bad = __builtin_amdgcn_readfirstlane(lds_load_asm(ctl + L_ABORT * 4)) != 0u;
    __amdgpu_buffer_rsrc_t ro = __builtin_amdgcn_make_buffer_rsrc(p.out, 0, 512u * 4096u * 4u, 0x00020000);
    const unsigned ob = ((unsigned)(rg * 128 + wave * 32 + (lane >> 4) * 4) * 4096u + G * 64 + (lane & 15) * 4) * 4u;
#pragma unroll
    for (int t = 0; t < 2; ++t)
#pragma unroll
        for (int r = 0; r < 4; ++r) {
            f32x4 y;
#pragma unroll
            for (int q = 0; q < 4; ++q) {
                const float v = acc[t][q][r] * scale;
                const float c = fminf(fmaxf(v, -60.f), 60.f);
                y[q] = bad ? __builtin_nanf("") : v * __builtin_amdgcn_rcpf(1.f + __expf(-c));
            }
            const u32x4 yy = __builtin_bit_cast(u32x4, y);
            const unsigned oo = ob + (unsigned)(t * 16 + r) * 16384u;
            __builtin_amdgcn_raw_buffer_store_b128(yy, ro, oo, 0, 16);
        }
}

extern "C" void kernel_launch(int rank, int nranks, ncclComm_t comm, hipStream_t stream, void* const* d_in, const int* in_sizes, int n_in,
                              void* d_out, int out_size, const int* in_dtypes, int out_dtype, void* const* d_out_all, void* const* d_ws_all, size_t ws_bytes) {
    if (nranks != NR || (unsigned)rank >= (unsigned)NR || ws_bytes < WS_NEED || n_in < 4 ||
        in_sizes[0] != 4096 * 512 || in_sizes[1] != 512 * 4096 || out_size != 512 * 4096) {
        fprintf(stderr, "kernel_launch: unexpected problem geometry\n");
        return;
    }
    Params p;
    p.x = (const uint8_t*)d_in[0]; p.w = (const uint8_t*)d_in[1]; p.sx = (const float*)d_in[2]; p.sw = (const float*)d_in[3];
    p.out = (float*)d_out;
    for (int r = 0; r < NR; ++r) p.ws[r] = (char*)d_ws_all[r];
    p.rank = rank;
    hipLaunchKernelGGL(fused_gemm_silu, dim3(NWG), dim3(NTHREADS), 0, stream, p);
}
```

```cpp
#include <hip/hip_runtime.h>
#include <rccl/rccl.h>
#include <cstdio>
#include <cstdint>

typedef unsigned u32x4 __attribute__((ext_vector_type(4)));
typedef unsigned u32x2 __attribute__((ext_vector_type(2)));
typedef int i32x8 __attribute__((ext_vector_type(8)));
typedef int i32x4 __attribute__((ext_vector_type(4)));
typedef float f32x4 __attribute__((ext_vector_type(4)));
typedef unsigned long long u64;

#define RING 16
constexpr int NR = 8;
constexpr int NWG = 256;
constexpr int NTHREADS = 704;
constexpr unsigned REC = 4096;
constexpr size_t XBUF = 2u << 20, X_OFF = 0;
constexpr size_t WBUF = 2u << 20, W_OFF = 4u << 20;
constexpr size_t PBUF = 2u << 20, P_OFF = 8u << 20;
constexpr size_t C_OFF = 12u << 20;
constexpr size_t CTR_OFF = C_OFF;
constexpr size_t WFLAG_OFF = C_OFF + (64u << 10);
constexpr size_t XFLAG_OFF = C_OFF + (96u << 10);
constexpr size_t TMO_OFF = C_OFF + (100u << 10);
constexpr size_t INV_OFF = C_OFF + (104u << 10);
constexpr size_t PROG_OFF = C_OFF + (108u << 10);
constexpr size_t WS_NEED = C_OFF + (1u << 20);
constexpr unsigned long long DEADLINE_TICKS = 50000000ull;
constexpr int PACE_D0 = 60, PACE_T1 = 30, PACE_T2 = 80, PACE_TPL = 2075;

constexpr unsigned L_STG = RING * 8192;
constexpr unsigned L_CODE = L_STG + 4 * 2304;
constexpr unsigned L_CTL = L_CODE + 256;
constexpr unsigned L_TOTAL = L_CTL + 64;
enum { L_CONS = 0  , L_DEC = 4  , L_READY = 8, L_XOWN = 9, L_XALL = 10, L_ABORT = 11, L_EXP = 12, L_START = 13, L_ANCH = 14, L_WORDS = 16 };

#define STAMP(i) do { } while (0)

struct Params {
    const uint8_t* x; const uint8_t* w; char* me; float* out; const float* sx; const float* sw;
    int rank, pad;
    char* pad2;
    char* ws[NR];
};
#define RLX_SYS __ATOMIC_RELAXED, __HIP_MEMORY_SCOPE_SYSTEM
#define RLX_AGT __ATOMIC_RELAXED, __HIP_MEMORY_SCOPE_AGENT
#define RLX_WG __ATOMIC_RELAXED, __HIP_MEMORY_SCOPE_WORKGROUP
#define CBAR() asm volatile("" ::: "memory")

__device__ __forceinline__ void glds_piece(unsigned voff, const char* sbase, unsigned ldsdst) {
    unsigned keep;
    asm volatile("s_mov_b32 %0, m0\n\ts_mov_b32 m0, %2\n\ts_nop 0\n\tglobal_load_lds_dwordx4 %1, %3\n\ts_mov_b32 m0, %0"
                 : "=&s"(keep) : "v"(voff), "s"(ldsdst), "s"(sbase) : "memory");
}
__device__ __forceinline__ void glds_piece_m(unsigned voff, const char* sbase, unsigned ldsdst, unsigned mlo) {
    unsigned keep;
    asm volatile("s_mov_b32 %0, m0\n\ts_mov_b32 m0, %2\n\ts_mov_b32 exec_lo, %4\n\ts_nop 1\n\tglobal_load_lds_dwordx4 %1, %3\n\ts_mov_b32 exec_lo, -1\n\ts_mov_b32 m0, %0"
                 : "=&s"(keep) : "v"(voff), "s"(ldsdst), "s"(sbase), "s"(mlo) : "memory");
}
__device__ __forceinline__ void lds_store_asm(unsigned addr, unsigned val) { asm volatile("ds_write_b32 %0, %1" :: "v"(addr), "v"(val) : "memory"); }
__device__ __forceinline__ unsigned lds_load_asm(unsigned addr) {
    unsigned v;
    asm volatile("ds_read_b32 %0, %1\n\ts_waitcnt lgkmcnt(0)" : "=&v"(v) : "v"(addr) : "memory");
    return v;
}
__device__ __forceinline__ unsigned lds_load4min_asm(unsigned addr) {
    u32x4 v;
    asm volatile("ds_read_b128 %0, %1\n\ts_waitcnt lgkmcnt(0)" : "=&v"(v) : "v"(addr) : "memory");
    const unsigned a = v.x < v.y ? v.x : v.y, c = v.z < v.w ? v.z : v.w;
    return a < c ? a : c;
}
__device__ __forceinline__ bool give_up(unsigned it, u64 t0, unsigned call, unsigned* tmo) {
    if ((it & 63u) == 0u) {
        if (__hip_atomic_load(tmo, RLX_AGT) == call || __builtin_amdgcn_s_memrealtime() - t0 > DEADLINE_TICKS) {
            __hip_atomic_store(tmo, call, RLX_AGT);
            return true;
        }
    }
    __builtin_amdgcn_s_sleep(1);
    return false;
}
__device__ __forceinline__ bool wait_flags32(const unsigned* f, bool active, unsigned call, unsigned* tmo) {
    const u64 t0 = __builtin_amdgcn_s_memrealtime();
    for (unsigned it = 1;; ++it) {
        const unsigned v = active ? __hip_atomic_load(f, RLX_SYS) : call;
        if (__all(v >= call)) return true;
        if (give_up(it, t0, call, tmo)) return false;
    }
}
__device__ __forceinline__ bool wait_flags64(const u64* f, bool active, unsigned call, unsigned* tmo, unsigned* hi) {
    const u64 t0 = __builtin_amdgcn_s_memrealtime();
    for (unsigned it = 1;; ++it) {
        const u64 v = active ? __hip_atomic_load(f, RLX_SYS) : (u64)call;
        if (__all((unsigned)v >= call)) { *hi = (unsigned)(v >> 32); return true; }
        if (give_up(it, t0, call, tmo)) { *hi = 0u; return false; }
    }
}
template <bool FOUR>
__device__ __forceinline__ void wait_lds(unsigned addr, unsigned need, unsigned abort_addr) {
    if (__builtin_amdgcn_readfirstlane(FOUR ? lds_load4min_asm(addr) : lds_load_asm(addr)) >= need) return;
    const u64 t0 = __builtin_amdgcn_s_memrealtime();
    for (;;) {
        __builtin_amdgcn_s_sleep(1);
        if (__builtin_amdgcn_readfirstlane(FOUR ? lds_load4min_asm(addr) : lds_load_asm(addr)) >= need) return;
        if (__builtin_amdgcn_s_memrealtime() - t0 > 2 * DEADLINE_TICKS) { lds_store_asm(abort_addr, 1u); return; }
    }
}
template <int CTRL, int RMASK>
__device__ __forceinline__ unsigned dpp0(unsigned x) { return (unsigned)__builtin_amdgcn_update_dpp(0, (int)x, CTRL, RMASK, 0xf, false); }

__device__ __forceinline__ unsigned wire_off(int q, int lane) {
    return lane >= 16 ? 512u + (unsigned)q * 768u + (unsigned)(lane - 16) * 16u
                      : ((q & 1) ? 3584u + (unsigned)(q >> 1) * 256u + (unsigned)lane * 16u
                                 : (unsigned)(lane >> 2) * 128u + (unsigned)(q >> 1) * 64u + (unsigned)(lane & 3) * 16u);
}
template <int N> __device__ __forceinline__ void drain(unsigned ctl, int pub) {
    asm volatile("s_waitcnt vmcnt(%0)" :: "n"(N) : "memory");
    if (224 - N > pub) lds_store_asm(ctl + L_READY * 4, 224u - N);
    if constexpr (N > 0) drain<N - 1>(ctl, pub);
}
template <int WIN>
__device__ __forceinline__ void unit8(const u64* ad, const unsigned* mk, unsigned voff, unsigned dst, unsigned ctl, int& issued, int& pub) {
#pragma unroll
    for (int c = 0; c < 8; ++c) {
        glds_piece_m(voff, (const char*)ad[c], dst + c * 1024, mk[c]);
        asm volatile("s_waitcnt vmcnt(%0)" :: "n"(WIN - 1) : "memory");
        ++issued;
        pub = max(pub, issued - WIN + 1);
        lds_store_asm(ctl + L_READY * 4, (unsigned)pub);
    }
}
template <int W0, int W1, int W2, int W3, int W4>
__device__ __forceinline__ void pull_loop(unsigned alo, unsigned ahi, unsigned mev, unsigned mod, unsigned lds0, unsigned ctl, int lane, int tau16) {
    int issued = 0, pub = 0;
    u64 ad[8];
    unsigned me_[8], mo_[8];
#pragma unroll 1
    for (int U = 0; U < 28; ++U) {
        const int q = U & 3;
        if (q == 0) {
#pragma unroll
            for (int c = 0; c < 8; ++c) {
                const int n = U * 2 + c;
                ad[c] = (u64)(unsigned)__builtin_amdgcn_readlane((int)alo, n) | ((u64)(unsigned)__builtin_amdgcn_readlane((int)ahi, n) << 32);
                me_[c] = (unsigned)__builtin_amdgcn_readlane((int)mev, n);
                mo_[c] = (unsigned)__builtin_amdgcn_readlane((int)mod, n);
            }
        }
        unsigned mk[8];
#pragma unroll
        for (int c = 0; c < 8; ++c) mk[c] = 0xffff0000u | (((q & 1) ? mo_[c] : me_[c]) >> ((q >> 1) * 16));
        const unsigned voff = wire_off(q, lane);
        const int J = 4 + U;
        const unsigned dst = lds0 + (unsigned)(J % RING) * 8192u;
        if (J >= RING) wait_lds<true>(ctl + L_CONS * 4, (unsigned)(J - RING + 1), ctl + L_ABORT * 4);
        const unsigned anch = __builtin_amdgcn_readfirstlane(lds_load_asm(ctl + L_ANCH * 4));
        const int lag = anch ? (int)((unsigned)__builtin_amdgcn_s_memrealtime() - anch) - (PACE_D0 + ((U * tau16) >> 4)) : 0;
        if (lag > PACE_T2) unit8<W4>(ad, mk, voff, dst, ctl, issued, pub);
        else if (lag > PACE_T1) unit8<W3>(ad, mk, voff, dst, ctl, issued, pub);
        else if (lag < -PACE_T2) unit8<W0>(ad, mk, voff, dst, ctl, issued, pub);
        else if (lag < -PACE_T1) unit8<W1>(ad, mk, voff, dst, ctl, issued, pub);
        else unit8<W2>(ad, mk, voff, dst, ctl, issued, pub);
    }
    drain<W4 - 2>(ctl, pub);
}

__global__ void __launch_bounds__(NTHREADS) fused_gemm_silu(Params p) {
    __shared__ __attribute__((aligned(16))) char lds[L_TOTAL];
    const int tid = threadIdx.x;
    const int lane = tid & 63;
    const int wave = __builtin_amdgcn_readfirstlane(tid >> 6);
    const int b = blockIdx.x;
    const int rank = p.rank;
    char* me = p.me;
    const uint8_t* px = p.x;
    const uint8_t* pw = p.w;
    char* dws = p.ws[(b + (b >> 3)) & 7];
    asm volatile("" :: "s"(rank), "s"(me), "s"(px), "s"(pw), "s"(dws));
    const unsigned lds0 = (unsigned)(size_t)lds;
    const unsigned ctl = lds0 + L_CTL;
    unsigned* tmo = (unsigned*)(me + TMO_OFF);
    const int idx = b >> 3;
    const int G = (b & 7) * 8 + (idx >> 2);
    const int rg = idx & 3;

    if (wave == 0) STAMP(0);
    if (tid < L_WORDS) ((unsigned*)(lds + L_CTL))[tid] = 0u;
    unsigned* ctr = (unsigned*)(me + CTR_OFF) + b * 16 + wave;
    unsigned cprev = __hip_atomic_load(ctr, RLX_AGT);

    if (wave >= 6 && wave < 10) {
        const int ew = wave - 6;
        const int Ge = b >> 2, kc = (b & 3) * 2 + (ew >> 1), hf = ew & 1;
        const int kh = lane & 1, il = (lane >> 1) & 3, g = (lane >> 3) & 3, qq = lane >> 5;
        const uint8_t* src = pw + (size_t)(kc * 64 + g * 16 + kh * 8) * 4096 + Ge * 64 + hf * 32 + qq * 16 + il * 4;
        unsigned v[8];
#pragma unroll
        for (int r = 0; r < 8; ++r) v[r] = *(const unsigned*)(src + (size_t)r * 4096);
        char* stg = lds + L_STG + ew * 2304;
        unsigned* hist = (unsigned*)(stg + 2048);
        hist[lane] = 0u;
        ((unsigned*)stg)[lane] = 0u;
        asm volatile("s_waitcnt lgkmcnt(0)\n\ts_barrier" ::: "memory");
#pragma unroll
        for (int r = 0; r < 3; ++r)
#pragma unroll
            for (int y = 0; y < 4; ++y)
                __hip_atomic_fetch_add(hist + ((v[r] >> (8 * y + 1)) & 63u), 1u, RLX_WG);
        unsigned t[4][2];
#pragma unroll
        for (int d = 0; d < 2; ++d) {
            const unsigned r0 = v[4 * d], r1 = v[4 * d + 1], r2 = v[4 * d + 2], r3 = v[4 * d + 3];
            const unsigned a0 = __builtin_amdgcn_perm(r1, r0, 0x05010400u), a1 = __builtin_amdgcn_perm(r1, r0, 0x07030602u);
            const unsigned a2 = __builtin_amdgcn_perm(r3, r2, 0x05010400u), a3 = __builtin_amdgcn_perm(r3, r2, 0x07030602u);
            t[0][d] = __builtin_amdgcn_perm(a2, a0, 0x05040100u);
            t[1][d] = __builtin_amdgcn_perm(a2, a0, 0x07060302u);
            t[2][d] = __builtin_amdgcn_perm(a3, a1, 0x05040100u);
            t[3][d] = __builtin_amdgcn_perm(a3, a1, 0x07060302u);
        }
        u32x4 o[2];
#pragma unroll
        for (int jj = 0; jj < 2; ++jj)
#pragma unroll
            for (int d = 0; d < 2; ++d) {
                const unsigned send = kh ? t[jj][d] : t[2 + jj][d], keep = kh ? t[2 + jj][d] : t[jj][d];
                const unsigned got = (unsigned)__builtin_amdgcn_update_dpp(0, (int)send, 0xB1  , 0xf, 0xf, false);
                o[jj][d] = kh ? got : keep;
                o[jj][2 + d] = kh ? keep : got;
            }
        const unsigned T = (unsigned)(Ge * 8 + kc);
        const unsigned call = __builtin_amdgcn_readfirstlane(cprev) + 1u;
        const unsigned buf = call & 1u;
        __amdgpu_buffer_rsrc_t rd = __builtin_amdgcn_make_buffer_rsrc(me + W_OFF + buf * WBUF, 0, (unsigned)WBUF, 0x00020000);
        unsigned sp[2];
#pragma unroll
        for (int jj = 0; jj < 2; ++jj) {
            const int j = kh * 2 + jj;
            sp[jj] = (unsigned)(j * 16 + ((g ^ j) * 4 + il));
            __builtin_amdgcn_raw_buffer_store_b128(o[jj], rd, T * 4096u + (unsigned)(hf * 2 + qq) * 1024u + sp[jj] * 16u, 0, 17);
        }
        unsigned pre = hist[lane];
        pre += dpp0<0x111, 0xf>(pre); pre += dpp0<0x112, 0xf>(pre); pre += dpp0<0x114, 0xf>(pre); pre += dpp0<0x118, 0xf>(pre);
        pre += dpp0<0x142, 0xa>(pre); pre += dpp0<0x143, 0xc>(pre);
        const unsigned lo = (unsigned)__builtin_amdgcn_ds_bpermute(((lane - 16) & 63) * 4, (int)pre);
        unsigned key = lane >= 15 ? (((pre - (lane >= 16 ? lo : 0u)) << 6) | (unsigned)(lane - 15)) : 0u;
        { unsigned z;
          z = dpp0<0x111, 0xf>(key); key = key > z ? key : z; z = dpp0<0x112, 0xf>(key); key = key > z ? key : z;
          z = dpp0<0x114, 0xf>(key); key = key > z ? key : z; z = dpp0<0x118, 0xf>(key); key = key > z ? key : z;
          z = dpp0<0x142, 0xa>(key); key = key > z ? key : z; z = dpp0<0x143, 0xc>(key); key = key > z ? key : z; }
        const unsigned base7 = ((unsigned)__builtin_amdgcn_readlane((int)key, 63) & 63u) * 2u;
        const unsigned K = (0x80u - base7) * 0x01010101u;
        unsigned e[4];
#pragma unroll
        for (int jj = 0; jj < 2; ++jj) {
            unsigned D[4], E[4];
#pragma unroll
            for (int d = 0; d < 4; ++d) {
                const unsigned xx = ((o[jj][d] & 0x7f7f7f7fu) + K) ^ 0x80808080u;
                D[d] = (xx & 0x1f1f1f1fu) | (o[jj][d] & 0x80808080u);
                E[d] = xx & 0xe0e0e0e0u;
            }
            u32x2 p01;
            p01.x = D[0] | ((D[3] << 5) & 0x60606060u);
            p01.y = D[1] | ((D[3] << 3) & 0x60606060u);
            const unsigned p2 = D[2] | ((D[3] << 1) & 0x20202020u) | ((D[3] >> 1) & 0x40404040u);
            *(u32x2*)(stg + qq * 1024 + 256 + sp[jj] * 8) = p01;
            *(unsigned*)(stg + qq * 1024 + 768 + sp[jj] * 4) = p2;
            e[jj * 2] = E[0] | (E[1] >> 4);
            e[jj * 2 + 1] = E[2] | (E[3] >> 4);
        }
        unsigned n = 0;
        while (n <= 127u && __ballot((e[0] | e[1] | e[2] | e[3]) != 0u) != 0ull) {
#pragma unroll
            for (int k = 0; k < 4; ++k) {
                const u64 m = __ballot(e[k] != 0u);
                const unsigned slot = n + 1u + __builtin_amdgcn_mbcnt_hi((unsigned)(m >> 32), __builtin_amdgcn_mbcnt_lo((unsigned)m, 0u));
                n += (unsigned)__builtin_popcountll(m);
                const unsigned nib4 = (unsigned)__builtin_ctz(e[k] | 0x80000000u) & 28u;
                const unsigned hhh = (e[k] >> (nib4 + 1u)) & 7u;
                const unsigned pos = (unsigned)qq * 1024u + sp[k >> 1] * 16u + (unsigned)((k & 1) * 8) + ((~nib4) & 4u) + (nib4 >> 3);
                if (e[k] != 0u && slot < 128u) *(unsigned short*)(stg + slot * 2u) = (unsigned short)(pos | (hhh << 12));
                e[k] &= ~(0xfu << nib4);
            }
        }
        const unsigned code = n > 127u ? 4u : (n >> 5);
        if (lane == 0) *(unsigned short*)stg = (unsigned short)base7;
        if (code >= 4u) {
            CBAR();
#pragma unroll
            for (int jj = 0; jj < 2; ++jj) *(u32x4*)(stg + qq * 1024 + sp[jj] * 16) = o[jj];
        }
        CBAR();
        __builtin_amdgcn_wave_barrier();
        {
            __amdgpu_buffer_rsrc_t rp = __builtin_amdgcn_make_buffer_rsrc(me + P_OFF + buf * PBUF + (size_t)T * REC, 0, REC, 0x00020000);
#pragma unroll
            for (int u = 0; u < 2; ++u) {
                const u32x4 tt = *(const u32x4*)(stg + u * 1024 + lane * 16);
                __builtin_amdgcn_raw_buffer_store_b128(tt, rp, wire_off(hf * 2 + u, lane), 0, 17);
            }
        }
        if (ew == 0) STAMP(3);
        asm volatile("s_waitcnt vmcnt(0)" ::: "memory");
        unsigned old = 0;
        if (lane == 0) old = __hip_atomic_fetch_add((unsigned*)(lds + L_CTL) + L_EXP, (1u << 16) | (code << (ew * 3)), RLX_WG);
        old = __builtin_amdgcn_readfirstlane(old) + ((1u << 16) | (code << (ew * 3)));
        if ((old >> 16) == 4u) {
            if (lane < NR)
                __hip_atomic_store((u64*)(p.ws[lane] + WFLAG_OFF) + buf * 2048 + Ge * 32 + rank * 4 + (b & 3), (u64)call | ((u64)(old & 0xfffu) << 32), RLX_SYS);
            STAMP(1);
        }
        if (lane == 0) __hip_atomic_store(ctr, call, RLX_AGT);

        unsigned codes = 0;
        unsigned done = 0;
        for (int ss = 0; ss < 7; ++ss) {
            unsigned ea[2][2], ev[2][2], Bv[2], cdd[2];
#pragma unroll
            for (int q = 0; q < 4; ++q) {
                const int J = 4 + ss * 4 + q;
#pragma unroll
                for (int cc = 0; cc < 2; ++cc) {
                    const int c = ew + cc * 4;
                    wait_lds<false>(ctl + L_READY * 4, (unsigned)((J - 4) * 8 + c + 1), ctl + L_ABORT * 4);
                    CBAR();
                    char* pb = lds + (J % RING) * 8192 + c * 1024;
                    if (q == 0) {
                        if (ss == 0 && cc == 0) codes = *(const unsigned*)(lds + L_CODE + (lane & 31) * 4);
                        const int nn = ss * 8 + c;
                        const int srcr = (rank + 1 + (nn + G) % 7) & 7, kcc = nn / 7;
                        cdd[cc] = (unsigned)__builtin_amdgcn_readlane((int)codes, srcr * 4 + (kcc >> 1)) >> ((kcc & 1) * 6);
                    }
                    const unsigned cd = (cdd[cc] >> ((q >> 1) * 3)) & 7u;
                    if (cd < 4u) {
                        const u32x2 p01 = *(const u32x2*)(pb + 256 + lane * 8);
                        const unsigned p2 = *(const unsigned*)(pb + 768 + lane * 4);
                        if ((q & 1) == 0) {
                            unsigned ex = *(const unsigned*)(pb + lane * 4);
                            Bv[cc] = ((unsigned)__builtin_amdgcn_readfirstlane(ex) & 0x7fu) * 0x01010101u;
                            if ((unsigned)lane >= (cd + 1u) * 16u) ex = 0u;
                            if (lane == 0) ex &= 0xffff0000u;
#pragma unroll
                            for (int h = 0; h < 2; ++h) {
                                const unsigned en = h ? (ex >> 16) : (ex & 0xffffu);
                                const int H = ((int)(en << 17)) >> 29;
                                ev[cc][h] = (unsigned)(H * 32) << ((en & 3u) * 8u);
                                ea[cc][h] = en & 0x7fcu;
                            }
                        }
                        asm volatile("s_waitcnt lgkmcnt(0)" ::: "memory");
                        u32x4 o4;
                        o4.x = (p01.x & 0x9f9f9f9fu) + Bv[cc];
                        o4.y = (p01.y & 0x9f9f9f9fu) + Bv[cc];
                        o4.z = (p2 & 0x9f9f9f9fu) + Bv[cc];
                        o4.w = (((p01.x >> 5) & 0x03030303u) | ((p01.y >> 3) & 0x0c0c0c0cu) | ((p2 >> 1) & 0x10101010u) | ((p2 << 1) & 0x80808080u)) + Bv[cc];
                        *(u32x4*)(pb + lane * 16) = o4;
#pragma unroll
                        for (int h = 0; h < 2; ++h)
                            if (ev[cc][h] != 0u && (ea[cc][h] >> 10) == (unsigned)(q & 1))
                                __hip_atomic_fetch_add((unsigned*)(pb + (ea[cc][h] & 0x3fcu)), ev[cc][h], RLX_WG);
                    }
                    CBAR();
                    lds_store_asm(ctl + (L_DEC + ew) * 4, ++done);
                }
            }
        }
        if (ew == 3) STAMP(10);
        return;
    } else if (wave == 1) {
        const int dest = (b + idx) & 7;
        const int xr = idx & 3, xs = (idx >> 2) & 3, xh = idx >> 4;
        const uint8_t* src = px + (size_t)(dest * 512 + xr * 128 + xh * 64 + (lane & 15)) * 512 + xs * 128 + (lane >> 4) * 16;
        u32x4 v[8];
#pragma unroll
        for (int t = 0; t < 4; ++t)
#pragma unroll
            for (int h = 0; h < 2; ++h) v[t * 2 + h] = *(const u32x4*)(src + (size_t)t * 16 * 512 + h * 64);
        asm volatile("s_waitcnt lgkmcnt(0)\n\ts_barrier" ::: "memory");
        const unsigned call = __builtin_amdgcn_readfirstlane(cprev) + 1u;
        __amdgpu_buffer_rsrc_t rd = __builtin_amdgcn_make_buffer_rsrc(dws + X_OFF + (call & 1u) * XBUF, 0, (unsigned)XBUF, 0x00020000);
        const unsigned obase = (unsigned)rank * 262144u + (unsigned)xs * 65536u + (unsigned)(xr * 8 + xh * 4) * 2048u + lane * 16u;
        asm volatile("s_waitcnt vmcnt(0)" ::: "memory");
#pragma unroll
        for (int u = 0; u < 8; ++u) {
            __builtin_amdgcn_raw_buffer_store_b128(v[u], rd, obase + (unsigned)u * 1024u, 0, 17);
            asm volatile("s_waitcnt vmcnt(0)" ::: "memory");
        }
        asm volatile("s_waitcnt vmcnt(0)" ::: "memory");
        if (lane == 0)
            __hip_atomic_store((unsigned*)(dws + XFLAG_OFF) + xr * 64 + (rank * 4 + xs) * 2 + xh, call, RLX_SYS);
        STAMP(2);
    } else {
        asm volatile("s_waitcnt lgkmcnt(0)\n\ts_barrier" ::: "memory");
    }
    const unsigned call = __builtin_amdgcn_readfirstlane(cprev) + 1u;
    if (lane == 0) __hip_atomic_store(ctr, call, RLX_AGT);
    const unsigned buf = call & 1u;

    if (wave == 4) {
        const unsigned xcc = __builtin_amdgcn_s_getreg(20 | (3 << 11)) & 15u;
        const int nsrc = (rank + 1 + (lane + G) % 7) & 7, nkc = (lane / 7) & 7;
        const u64 ra = (u64)(p.ws[nsrc] + P_OFF + (size_t)buf * PBUF + (size_t)(G * 8 + nkc) * REC);
        const u64* pf = lane == 32 ? (const u64*)(me + INV_OFF) + xcc * 16 : (const u64*)(me + WFLAG_OFF) + buf * 2048 + G * 32 + (lane & 31);
        const bool act = lane < 32 && (lane >> 2) != rank;
        bool ok = true;
        u64 fv;
        {
            const u64 t0 = __builtin_amdgcn_s_memrealtime();
            for (unsigned it = 1;; ++it) {
                fv = (act || lane == 32) ? __hip_atomic_load(pf, RLX_SYS) : 0ull;
                if (__all(!act || (unsigned)fv >= call)) break;
                if (give_up(it, t0, call, tmo)) { ok = false; break; }
            }
        }
        STAMP(5);
        if (ok) {
            const unsigned codes = (unsigned)(fv >> 32);
            lds_store_asm(lds0 + L_CODE + lane * 4, codes);
            if ((unsigned)__builtin_amdgcn_readlane((int)(unsigned)fv, 32) != call) __builtin_amdgcn_fence(__ATOMIC_ACQUIRE, "");
            const unsigned cd = (unsigned)__builtin_amdgcn_ds_bpermute((nsrc * 4 + (nkc >> 1)) * 4, (int)codes) >> ((nkc & 1) * 6);
            const unsigned ca = cd & 7u, cb2 = (cd >> 3) & 7u;
            const unsigned mev = (ca >= 4u ? 0xffffu : ((16u << (4u * ca)) - 1u)) | ((cb2 >= 4u ? 0xffffu : ((16u << (4u * cb2)) - 1u)) << 16);
            const unsigned mod = (ca >= 4u ? 0xffffu : 0u) | (cb2 >= 4u ? 0xffff0000u : 0u);
            lds_store_asm(ctl + L_START * 4, (unsigned)__builtin_amdgcn_s_memrealtime() | 1u);
            unsigned ln = lane < 56 ? 24u + (ca >= 4u ? 2u : 0u) + (cb2 >= 4u ? 2u : 0u) + ((ca | cb2) >= 4u ? 4u : max(ca, cb2) + 1u) : 0u;
            ln += dpp0<0x111, 0xf>(ln); ln += dpp0<0x112, 0xf>(ln); ln += dpp0<0x114, 0xf>(ln); ln += dpp0<0x118, 0xf>(ln);
            ln += dpp0<0x142, 0xa>(ln); ln += dpp0<0x143, 0xc>(ln);
            const int tau16 = (__builtin_amdgcn_readlane((int)ln, 63) * PACE_TPL) / 1750;
            asm volatile("s_waitcnt vmcnt(0)" ::: "memory");
            STAMP(6);
            pull_loop<6, 8, 10, 12, 15>((unsigned)ra, (unsigned)(ra >> 32), mev, mod, lds0, ctl, lane, tau16);
        }
        if (!ok) { lds_store_asm(ctl + L_ABORT * 4, 1u); lds_store_asm(ctl + L_READY * 4, 224u); lds_store_asm(ctl + L_START * 4, 1u); }
        STAMP(7);
        return;
    }
    if (wave == 10) {
        wait_lds<false>(ctl + L_START * 4, 1u, ctl + L_ABORT * 4);
        const unsigned st = lds_load_asm(ctl + L_START * 4);
        if (lane == 0) {
            const u64 key = ((u64)call << 32) | (u64)(0xffffffffu - st);
            const u64 old = __hip_atomic_fetch_max((u64*)(me + PROG_OFF), key, RLX_AGT);
            const u64 best = old > key ? old : key;
            unsigned an = 0xffffffffu - (unsigned)best;
            if (st - an > 100000u) an = st;
            lds_store_asm(ctl + L_ANCH * 4, an | 1u);
        }
        return;
    }
    if (wave == 5) {
        const unsigned* xf = (const unsigned*)(me + XFLAG_OFF) + rg * 64 + lane;
        unsigned dummy;
        if (b < 8) {
            __builtin_amdgcn_fence(__ATOMIC_ACQUIRE, "");
            asm volatile("s_waitcnt vmcnt(0)" ::: "memory");
            const unsigned xcc = __builtin_amdgcn_s_getreg(20 | (3 << 11)) & 15u;
            if (lane == 0) __hip_atomic_store((unsigned*)(me + INV_OFF) + xcc * 32, call, RLX_AGT);
        }
        bool ok = wait_flags64((const u64*)(me + WFLAG_OFF) + buf * 2048 + G * 32 + rank * 4 + (lane & 3), lane < 4, call, tmo, &dummy);
        if (ok) {
            const char* sb = me + W_OFF + (size_t)buf * WBUF + (size_t)G * 32768;
            for (int q = 0; q < 4; ++q)
#pragma unroll
                for (int c = 0; c < 8; ++c) glds_piece(lane * 16u, sb + c * 4096 + q * 1024, lds0 + q * 8192 + c * 1024);
            asm volatile("s_waitcnt vmcnt(0)" ::: "memory");
            ok = wait_flags32(xf, (lane >> 3) == rank, call, tmo);
        }
        if (!ok) lds_store_asm(ctl + L_ABORT * 4, 1u);
        lds_store_asm(ctl + L_XOWN * 4, 1u);
        STAMP(8);
        if (ok) ok = wait_flags32(xf, true, call, tmo);
        if (!ok) lds_store_asm(ctl + L_ABORT * 4, 1u);
        lds_store_asm(ctl + L_XALL * 4, 1u);
        STAMP(9);
        return;
    }

    const float scale = p.sx[0] * p.sw[0];
    f32x4 acc[2][4];
#pragma unroll
    for (int t = 0; t < 2; ++t)
#pragma unroll
        for (int q = 0; q < 4; ++q) acc[t][q] = (f32x4){0.f, 0.f, 0.f, 0.f};
    const char* xb = me + X_OFF + (size_t)buf * XBUF + (size_t)(rg * 8 + wave * 2) * 2048 + lane * 16;
    i32x4 A[8][2];
    int pi = G % 7, pk = 0, pc = 0;
    auto loadA1 = [&](int c) {
        const char* a = xb + (size_t)((rank + 1 + pi) & 7) * 262144 + (size_t)((pk >> 1) & 3) * 65536 + (pk & 1) * 1024;
        A[c][0] = *(const i32x4*)a;
        A[c][1] = *(const i32x4*)(a + 2048);
        if (++pi == 7) pi = 0;
        if (++pc == 7) { pc = 0; ++pk; }
    };
    const unsigned wofs = (unsigned)(((lane & 3) * 16) + (((lane >> 4) ^ (lane & 3)) * 4) + ((lane >> 2) & 3)) * 16u;
    __amdgpu_buffer_rsrc_t ro = __builtin_amdgcn_make_buffer_rsrc(p.out, 0, 512u * 4096u * 4u, 0x00020000);
    const unsigned ob = ((unsigned)(rg * 128 + wave * 32 + (lane & 15)) * 4096u + G * 64 + (lane >> 4) * 4) * 4u;
    auto sstep = [&](int J0, int MODE) {
#pragma unroll
        for (int q = 0; q < 4; ++q) {
            const int J = J0 + q;
            const char* bs = lds + (J % RING) * 8192 + wofs;
#pragma unroll
            for (int hh = 0; hh < 2; ++hh) {
                if (MODE != 0) { wait_lds<true>(ctl + L_DEC * 4, (unsigned)((J - 4) * 2 + hh + 1), ctl + L_ABORT * 4); CBAR(); }
#pragma unroll
                for (int c = hh * 4; c < hh * 4 + 4; c += 2) {
                    const i32x4 lo = *(const i32x4*)(bs + c * 1024), hi = *(const i32x4*)(bs + (c + 1) * 1024);
                    const i32x8 W = __builtin_shufflevector(lo, hi, 0, 1, 2, 3, 4, 5, 6, 7);
#pragma unroll
                    for (int t = 0; t < 2; ++t) {
                        const i32x8 a = __builtin_shufflevector(A[c][t], A[c + 1][t], 0, 1, 2, 3, 4, 5, 6, 7);
                        acc[t][q] = __builtin_amdgcn_mfma_scale_f32_16x16x128_f8f6f4(W, a, acc[t][q], 1, 1, 0, 0x7f7f7f7f, 0, 0x7f7f7f7f);
                    }
                }
                asm volatile("" : "+v"(acc[0][q]), "+v"(acc[1][q]) :: "memory");
                if (MODE == 1 && q == 3) {
#pragma unroll
                    for (int c = hh * 4; c < hh * 4 + 4; ++c) loadA1(c);
                }
            }
            if (MODE != 2) {
                lds_store_asm(ctl + (L_CONS + wave) * 4, (unsigned)(J + 1));
            } else {
                const bool bad = __builtin_amdgcn_readfirstlane(lds_load_asm(ctl + L_ABORT * 4)) != 0u;
#pragma unroll
                for (int t = 0; t < 2; ++t) {
                    f32x4 y;
#pragma unroll
                    for (int r = 0; r < 4; ++r) {
                        const float v = acc[t][q][r] * scale;
                        const float cl = fminf(fmaxf(v, -60.f), 60.f);
                        y[r] = bad ? __builtin_nanf("") : v * __builtin_amdgcn_rcpf(1.f + __expf(-cl));
                    }
                    __builtin_amdgcn_raw_buffer_store_b128(__builtin_bit_cast(u32x4, y), ro, ob + (unsigned)t * 16u * 16384u + (unsigned)q * 64u, 0, 16);
                }
                CBAR();
            }
        }
    };
    wait_lds<false>(ctl + L_XOWN * 4, 1u, ctl + L_ABORT * 4);
    CBAR();
#pragma unroll
    for (int c = 0; c < 8; ++c) {
        const char* a = xb + (size_t)rank * 262144 + (size_t)(c >> 1) * 65536 + (c & 1) * 1024;
        A[c][0] = *(const i32x4*)a;
        A[c][1] = *(const i32x4*)(a + 2048);
    }
    sstep(0, 0);
    wait_lds<false>(ctl + L_XALL * 4, 1u, ctl + L_ABORT * 4);
    CBAR();
    if (wave == 3) STAMP(11);
#pragma unroll
    for (int c = 0; c < 8; ++c) loadA1(c);
    for (int J0 = 4; J0 < 28; J0 += 4) sstep(J0, 1);
    sstep(28, 2);
    if (wave == 3) STAMP(12);
}

extern "C" void kernel_launch(int rank, int nranks, ncclComm_t comm, hipStream_t stream, void* const* d_in, const int* in_sizes, int n_in,
                              void* d_out, int out_size, const int* in_dtypes, int out_dtype, void* const* d_out_all, void* const* d_ws_all, size_t ws_bytes) {
    if (nranks != NR || (unsigned)rank >= (unsigned)NR || ws_bytes < WS_NEED || n_in < 4 ||
        in_sizes[0] != 4096 * 512 || in_sizes[1] != 512 * 4096 || out_size != 512 * 4096) {
        fprintf(stderr, "kernel_launch: unexpected problem geometry\n");
        return;
    }
    Params p;
    p.x = (const uint8_t*)d_in[0]; p.w = (const uint8_t*)d_in[1]; p.sx = (const float*)d_in[2]; p.sw = (const float*)d_in[3];
    p.out = (float*)d_out;
    for (int r = 0; r < NR; ++r) p.ws[r] = (char*)d_ws_all[r];
    p.rank = rank; p.me = (char*)d_ws_all[rank]; p.pad = 0; p.pad2 = nullptr;
    hipLaunchKernelGGL(fused_gemm_silu, dim3(NWG), dim3(NTHREADS), 0, stream, p);
}
```

```cpp
#include <hip/hip_runtime.h>
#include <rccl/rccl.h>
#include <cstdio>
#include <cstdint>

typedef unsigned u32x4 __attribute__((ext_vector_type(4)));
typedef unsigned u32x2 __attribute__((ext_vector_type(2)));
typedef int i32x8 __attribute__((ext_vector_type(8)));
typedef int i32x4 __attribute__((ext_vector_type(4)));
typedef float f32x4 __attribute__((ext_vector_type(4)));
typedef unsigned long long u64;

constexpr int NR = 8;
constexpr int NWG = 256;
constexpr int NTHREADS = 704;
constexpr int RING = 16;
constexpr unsigned REC = 4096;
constexpr size_t XBUF = 2u << 20, X_OFF = 0;
constexpr size_t WBUF = 2u << 20, W_OFF = 4u << 20;
constexpr size_t PBUF = 2u << 20, P_OFF = 8u << 20;
constexpr size_t C_OFF = 12u << 20;
constexpr size_t CTR_OFF = C_OFF;
constexpr size_t WFLAG_OFF = C_OFF + (64u << 10);
constexpr size_t XFLAG_OFF = C_OFF + (96u << 10);
constexpr size_t TMO_OFF = C_OFF + (100u << 10);
constexpr size_t INV_OFF = C_OFF + (104u << 10);
constexpr size_t ANCH_OFF = C_OFF + (108u << 10);
constexpr size_t WS_NEED = C_OFF + (1u << 20);
constexpr u64 DEADLINE_TICKS = 95000000ull;
constexpr int PACE_D0 = 100, PACE_T1 = 30, PACE_T2 = 80;
constexpr int PACE_TPL = 2040;
constexpr int POLL_START = 300;

constexpr unsigned L_STG = RING * 8192;
constexpr unsigned L_CODE = L_STG + 4 * 2304;
constexpr unsigned L_CTL = L_CODE + 256;
constexpr unsigned L_TOTAL = L_CTL + 64;
enum { L_CONS = 0  , L_DEC = 4  , L_READY = 8, L_XOWN = 9, L_XALL = 10, L_ABORT = 11, L_EXP = 12, L_START = 13,
       L_ANCH = 14, L_TAU = 15, L_WORDS = 16 };

struct Params {
    const uint8_t* x; const uint8_t* w; char* me; float* out; const float* sx; const float* sw;
    int rank, pad;
    char* pad2;
    char* ws[NR];
};
#define RLX_SYS __ATOMIC_RELAXED, __HIP_MEMORY_SCOPE_SYSTEM
#define RLX_AGT __ATOMIC_RELAXED, __HIP_MEMORY_SCOPE_AGENT
#define RLX_WG __ATOMIC_RELAXED, __HIP_MEMORY_SCOPE_WORKGROUP
#define CBAR() asm volatile("" ::: "memory")
#define VM_DRAIN() asm volatile("s_waitcnt vmcnt(0)" ::: "memory")
#define WG_BARRIER(c) do { asm volatile("s_waitcnt lgkmcnt(0)\n\ts_barrier" ::: "memory"); asm volatile("" : "+v"(c)); } while (0)
constexpr int AUX_SYS = 17, AUX_AGT = 16;

__device__ __forceinline__ void glds_piece(unsigned voff, const char* sbase, unsigned ldsdst, unsigned mlo) {
    unsigned keep;
    asm volatile("s_mov_b32 %0, m0\n\ts_mov_b32 m0, %2\n\ts_mov_b32 exec_lo, %4\n\ts_nop 1\n\t"
                 "global_load_lds_dwordx4 %1, %3\n\ts_mov_b32 exec_lo, -1\n\ts_mov_b32 m0, %0"
                 : "=&s"(keep) : "v"(voff), "s"(ldsdst), "s"(sbase), "s"(mlo) : "memory");
}
__device__ __forceinline__ void lds_st(unsigned addr, unsigned val) { asm volatile("ds_write_b32 %0, %1" :: "v"(addr), "v"(val) : "memory"); }
__device__ __forceinline__ unsigned lds_ld(unsigned addr) {
    unsigned v;
    asm volatile("ds_read_b32 %0, %1\n\ts_waitcnt lgkmcnt(0)" : "=&v"(v) : "v"(addr) : "memory");
    return v;
}
__device__ __forceinline__ unsigned lds_ld4min(unsigned addr) {
    u32x4 v;
    asm volatile("ds_read_b128 %0, %1\n\ts_waitcnt lgkmcnt(0)" : "=&v"(v) : "v"(addr) : "memory");
    const unsigned a = v.x < v.y ? v.x : v.y, c = v.z < v.w ? v.z : v.w;
    return a < c ? a : c;
}
__device__ __forceinline__ unsigned uni(unsigned v) { return __builtin_amdgcn_readfirstlane(v); }
__device__ __forceinline__ void sleep_until(u64 t) { while ((long long)(__builtin_amdgcn_s_memrealtime() - t) < 0) __builtin_amdgcn_s_sleep(2); }
template <int NAP>
__device__ __forceinline__ bool give_up(unsigned it, u64 t0, unsigned call, unsigned* tmo) {
    __builtin_amdgcn_s_sleep(NAP);
    if ((it & 63u) == 0u) {
        if (uni(__hip_atomic_load(tmo, RLX_AGT)) == call || __builtin_amdgcn_s_memrealtime() - t0 > DEADLINE_TICKS) {
            __hip_atomic_store(tmo, call, RLX_AGT);
            return true;
        }
    }
    return false;
}
template <int NAP>
__device__ __forceinline__ bool wait_flags32(const unsigned* f, bool active, unsigned call, unsigned* tmo) {
    const u64 t0 = __builtin_amdgcn_s_memrealtime();
    for (unsigned it = 1;; ++it) {
        const unsigned v = active ? __hip_atomic_load(f, RLX_SYS) : call;
        if (__all(v >= call)) return true;
        if (give_up<NAP>(it, t0, call, tmo)) return false;
    }
}
template <int NAP>
__device__ __forceinline__ bool wait_flags64(const u64* f, bool need, bool read, unsigned call, unsigned* tmo, u64* out) {
    const u64 t0 = __builtin_amdgcn_s_memrealtime();
    u64 v = 0ull;
    for (unsigned it = 1;; ++it) {
        if ((need || read) && (unsigned)v < call) v = __hip_atomic_load(f, RLX_SYS);
        if (__all(!need || (unsigned)v >= call)) { *out = v; return true; }
        if (give_up<NAP>(it, t0, call, tmo)) { *out = 0ull; return false; }
    }
}
template <bool FOUR>
__device__ __forceinline__ unsigned wait_lds(unsigned addr, unsigned need, unsigned abort_addr) {
    unsigned v = uni(FOUR ? lds_ld4min(addr) : lds_ld(addr));
    if (v >= need) return v;
    const u64 t0 = __builtin_amdgcn_s_memrealtime();
    for (unsigned it = 1;; ++it) {
        __builtin_amdgcn_s_sleep(1);
        v = uni(FOUR ? lds_ld4min(addr) : lds_ld(addr));
        if (v >= need) return v;
        if ((it & 1023u) == 0u && (uni(lds_ld(abort_addr)) != 0u || __builtin_amdgcn_s_memrealtime() - t0 > 2 * DEADLINE_TICKS)) {
            lds_st(abort_addr, 1u);
            return need;
        }
    }
}
template <int CTRL, int RMASK>
__device__ __forceinline__ unsigned dpp0(unsigned x) { return (unsigned)__builtin_amdgcn_update_dpp(0, (int)x, CTRL, RMASK, 0xf, false); }
__device__ __forceinline__ unsigned wave_scan(unsigned x) {
    x += dpp0<0x111, 0xf>(x); x += dpp0<0x112, 0xf>(x); x += dpp0<0x114, 0xf>(x); x += dpp0<0x118, 0xf>(x);
    x += dpp0<0x142, 0xa>(x); x += dpp0<0x143, 0xc>(x);
    return x;
}
__device__ __forceinline__ unsigned wave_max63(unsigned x) {
    x = max(x, dpp0<0x111, 0xf>(x)); x = max(x, dpp0<0x112, 0xf>(x)); x = max(x, dpp0<0x114, 0xf>(x)); x = max(x, dpp0<0x118, 0xf>(x));
    x = max(x, dpp0<0x142, 0xa>(x)); x = max(x, dpp0<0x143, 0xc>(x));
    return x;
}
__device__ __forceinline__ unsigned wire_off(int q, int lane) {
    return lane >= 16 ? 512u + (unsigned)q * 768u + (unsigned)(lane - 16) * 16u
                      : ((q & 1) ? 3584u + (unsigned)(q >> 1) * 256u + (unsigned)lane * 16u
                                 : (unsigned)(lane >> 2) * 128u + (unsigned)(q >> 1) * 64u + (unsigned)(lane & 3) * 16u);
}
__device__ __forceinline__ unsigned mask_even(unsigned cd) {
    const unsigned ca = cd & 7u, cb = (cd >> 3) & 7u;
    return (ca >= 4u ? 0xffffu : ((16u << (4u * ca)) - 1u)) | ((cb >= 4u ? 0xffffu : ((16u << (4u * cb)) - 1u)) << 16);
}
__device__ __forceinline__ unsigned mask_odd(unsigned cd) { return ((cd & 4u) ? 0xffffu : 0u) | ((cd & 32u) ? 0xffff0000u : 0u); }

template <int N> __device__ __forceinline__ void drain(unsigned ctl, int pub) {
    asm volatile("s_waitcnt vmcnt(%0)" :: "n"(N) : "memory");
    if (224 - N > pub) lds_st(ctl + L_READY * 4, 224u - N);
    if constexpr (N > 0) drain<N - 1>(ctl, pub);
}
template <int WIN>
__device__ __forceinline__ void unit8(const u64* ad, const unsigned* mk, unsigned voff, unsigned dst, unsigned ctl, int& issued, int& pub) {
#pragma unroll
    for (int c = 0; c < 8; ++c) {
        glds_piece(voff, (const char*)ad[c], dst + c * 1024, mk[c]);
        asm volatile("s_waitcnt vmcnt(%0)" :: "n"(WIN - 1) : "memory");
        ++issued;
        pub = max(pub, issued - WIN + 1);
        lds_st(ctl + L_READY * 4, (unsigned)pub);
    }
}

__global__ void __launch_bounds__(NTHREADS) fused_gemm_silu(Params p) {
    __shared__ __attribute__((aligned(16))) char lds[L_TOTAL];
    const int tid = threadIdx.x;
    const int lane = tid & 63;
    const int wave = __builtin_amdgcn_readfirstlane(tid >> 6);
    const int b = blockIdx.x;
    const int rank = p.rank;
    char* me = p.me;
    const uint8_t* px = p.x;
    const uint8_t* pw = p.w;
    char* dws = p.ws[(b + (b >> 3)) & 7];
    asm volatile("" :: "s"(rank), "s"(me), "s"(px), "s"(pw), "s"(dws));
    const unsigned lds0 = (unsigned)(size_t)lds;
    const unsigned ctl = lds0 + L_CTL;
    const unsigned abortw = ctl + L_ABORT * 4;
    unsigned* tmo = (unsigned*)(me + TMO_OFF);
    const int idx = b >> 3;
    const int G = (b & 7) * 8 + (idx >> 2);
    const int rg = idx & 3;
    if (tid < L_WORDS) ((unsigned*)(lds + L_CTL))[tid] = (tid >= L_DEC && tid < L_DEC + 4) ? 8u : 0u;
    unsigned* ctr = (unsigned*)(me + CTR_OFF) + b * 16 + wave;
    unsigned call;
    const u64 tstart = __builtin_amdgcn_s_memrealtime();

    if (wave >= 6 && wave < 10) {
        const int ew = wave - 6;
        const int Ge = b >> 2, kc = (b & 3) * 2 + (ew >> 1), hf = ew & 1;
        const int kh = lane & 1, il = (lane >> 1) & 3, g = (lane >> 3) & 3, qq = lane >> 5;
        const uint8_t* src = pw + (size_t)(kc * 64 + g * 16 + kh * 8) * 4096 + Ge * 64 + hf * 32 + qq * 16 + il * 4;
        unsigned cprev = __hip_atomic_load(ctr, RLX_AGT);
        unsigned v[8];
#pragma unroll
        for (int r = 0; r < 8; ++r) v[r] = *(const unsigned*)(src + (size_t)r * 4096);
        char* stg = lds + L_STG + ew * 2304;
        unsigned* hist = (unsigned*)(stg + 2048);
        hist[lane] = 0u;
        ((unsigned*)stg)[lane] = 0u;
        WG_BARRIER(cprev);
#pragma unroll
        for (int r = 0; r < 3; ++r)
#pragma unroll
            for (int y = 0; y < 4; ++y) __hip_atomic_fetch_add(hist + ((v[r] >> (8 * y + 1)) & 63u), 1u, RLX_WG);
        unsigned t[4][2];
#pragma unroll
        for (int d = 0; d < 2; ++d) {
            const unsigned r0 = v[4 * d], r1 = v[4 * d + 1], r2 = v[4 * d + 2], r3 = v[4 * d + 3];
            const unsigned a0 = __builtin_amdgcn_perm(r1, r0, 0x05010400u), a1 = __builtin_amdgcn_perm(r1, r0, 0x07030602u);
            const unsigned a2 = __builtin_amdgcn_perm(r3, r2, 0x05010400u), a3 = __builtin_amdgcn_perm(r3, r2, 0x07030602u);
            t[0][d] = __builtin_amdgcn_perm(a2, a0, 0x05040100u);
            t[1][d] = __builtin_amdgcn_perm(a2, a0, 0x07060302u);
            t[2][d] = __builtin_amdgcn_perm(a3, a1, 0x05040100u);
            t[3][d] = __builtin_amdgcn_perm(a3, a1, 0x07060302u);
        }
        u32x4 o[2];
#pragma unroll
        for (int jj = 0; jj < 2; ++jj)
#pragma unroll
            for (int d = 0; d < 2; ++d) {
                const unsigned send = kh ? t[jj][d] : t[2 + jj][d], keep = kh ? t[2 + jj][d] : t[jj][d];
                const unsigned got = (unsigned)__builtin_amdgcn_update_dpp(0, (int)send, 0xB1  , 0xf, 0xf, false);
                o[jj][d] = kh ? got : keep;
                o[jj][2 + d] = kh ? keep : got;
            }
        const unsigned T = (unsigned)(Ge * 8 + kc);
        call = uni(cprev) + 1u;
        const unsigned buf = call & 1u;
        __amdgpu_buffer_rsrc_t rd = __builtin_amdgcn_make_buffer_rsrc(me + W_OFF + buf * WBUF, 0, (unsigned)WBUF, 0x00020000);
        unsigned sp[2];
#pragma unroll
        for (int jj = 0; jj < 2; ++jj) {
            const int j = kh * 2 + jj;
            sp[jj] = (unsigned)(j * 16 + ((g ^ j) * 4 + il));
            __builtin_amdgcn_raw_buffer_store_b128(o[jj], rd, T * 4096u + (unsigned)(hf * 2 + qq) * 1024u + sp[jj] * 16u, 0, AUX_AGT);
        }
        const unsigned pre = wave_scan(hist[lane]);
        const unsigned lo = (unsigned)__builtin_amdgcn_ds_bpermute(((lane - 16) & 63) * 4, (int)pre);
        const unsigned key = wave_max63(lane >= 15 ? (((pre - (lane >= 16 ? lo : 0u)) << 6) | (unsigned)(lane - 15)) : 0u);
        const unsigned base7 = ((unsigned)__builtin_amdgcn_readlane((int)key, 63) & 63u) * 2u;
        const unsigned K = (0x80u - base7) * 0x01010101u;
        unsigned e[4];
#pragma unroll
        for (int jj = 0; jj < 2; ++jj) {
            unsigned D[4], E[4];
#pragma unroll
            for (int d = 0; d < 4; ++d) {
                const unsigned xx = ((o[jj][d] & 0x7f7f7f7fu) + K) ^ 0x80808080u;
                D[d] = (xx & 0x1f1f1f1fu) | (o[jj][d] & 0x80808080u);
                E[d] = xx & 0xe0e0e0e0u;
            }
            u32x2 p01;
            p01.x = D[0] | ((D[3] << 5) & 0x60606060u);
            p01.y = D[1] | ((D[3] << 3) & 0x60606060u);
            const unsigned p2 = D[2] | ((D[3] << 1) & 0x20202020u) | ((D[3] >> 1) & 0x40404040u);
            *(u32x2*)(stg + qq * 1024 + 256 + sp[jj] * 8) = p01;
            *(unsigned*)(stg + qq * 1024 + 768 + sp[jj] * 4) = p2;
            e[jj * 2] = E[0] | (E[1] >> 4);
            e[jj * 2 + 1] = E[2] | (E[3] >> 4);
        }
        unsigned n = 0;
        while (n <= 127u && __ballot((e[0] | e[1] | e[2] | e[3]) != 0u) != 0ull) {
#pragma unroll
            for (int k = 0; k < 4; ++k) {
                const u64 m = __ballot(e[k] != 0u);
                const unsigned slot = n + 1u + __builtin_amdgcn_mbcnt_hi((unsigned)(m >> 32), __builtin_amdgcn_mbcnt_lo((unsigned)m, 0u));
                n += (unsigned)__builtin_popcountll(m);
                const unsigned nib4 = (unsigned)__builtin_ctz(e[k] | 0x80000000u) & 28u;
                const unsigned hhh = (e[k] >> (nib4 + 1u)) & 7u;
                const unsigned pos = (unsigned)qq * 1024u + sp[k >> 1] * 16u + (unsigned)((k & 1) * 8) + ((~nib4) & 4u) + (nib4 >> 3);
                if (e[k] != 0u && slot < 128u) *(unsigned short*)(stg + slot * 2u) = (unsigned short)(pos | (hhh << 12));
                e[k] &= ~(0xfu << nib4);
            }
        }
        const unsigned code = n > 127u ? 4u : (n >> 5);
        if (lane == 0) *(unsigned short*)stg = (unsigned short)base7;
        if (code >= 4u) {
            CBAR();
#pragma unroll
            for (int jj = 0; jj < 2; ++jj) *(u32x4*)(stg + qq * 1024 + sp[jj] * 16) = o[jj];
        }
        CBAR();
        __builtin_amdgcn_wave_barrier();
        {
            __amdgpu_buffer_rsrc_t rp = __builtin_amdgcn_make_buffer_rsrc(me + P_OFF + buf * PBUF + (size_t)T * REC, 0, REC, 0x00020000);
#pragma unroll
            for (int u = 0; u < 2; ++u) {
                const u32x4 tt = *(const u32x4*)(stg + u * 1024 + lane * 16);
                __builtin_amdgcn_raw_buffer_store_b128(tt, rp, wire_off(hf * 2 + u, lane), 0, AUX_SYS);
            }
        }
        VM_DRAIN();
        unsigned old = 0;
        if (lane == 0) old = __hip_atomic_fetch_add((unsigned*)(lds + L_CTL) + L_EXP, (1u << 16) | (code << (ew * 3)), RLX_WG);
        old = uni(old) + ((1u << 16) | (code << (ew * 3)));
        if ((old >> 16) == 4u) {
            if (lane < NR)
                __hip_atomic_store((u64*)(p.ws[lane] + WFLAG_OFF) + buf * 2048 + Ge * 32 + rank * 4 + (b & 3),
                                   (u64)call | ((u64)(old & 0xfffu) << 32), RLX_SYS);
        }
        if (lane == 0) __hip_atomic_store(ctr, call, RLX_AGT);

        unsigned ea[2][2] = {}, ev[2][2] = {}, Bv[2] = {}, cdd[2] = {}, rdy = 0u;
#pragma unroll 1
        for (int U = 0; U < 28; ++U) {
            const int q = U & 3;
#pragma unroll
            for (int cc = 0; cc < 2; ++cc) {
                const int c = ew + cc * 4;
                if (rdy < (unsigned)(U * 8 + c + 1)) rdy = wait_lds<false>(ctl + L_READY * 4, (unsigned)(U * 8 + c + 1), abortw);
                CBAR();
                char* pb = lds + ((U + 4) % RING) * 8192 + c * 1024;
                if (q == 0) cdd[cc] = uni(*(const unsigned*)(lds + L_CODE + (U * 2 + c) * 4));
                const unsigned cd = (cdd[cc] >> ((q >> 1) * 3)) & 7u;
                if (cd < 4u) {
                    const u32x2 p01 = *(const u32x2*)(pb + 256 + lane * 8);
                    const unsigned p2 = *(const unsigned*)(pb + 768 + lane * 4);
                    if ((q & 1) == 0) {
                        unsigned ex = *(const unsigned*)(pb + lane * 4);
                        Bv[cc] = (uni(ex) & 0x7fu) * 0x01010101u;
                        if ((unsigned)lane >= (cd + 1u) * 16u) ex = 0u;
                        if (lane == 0) ex &= 0xffff0000u;
#pragma unroll
                        for (int h = 0; h < 2; ++h) {
                            const unsigned en = h ? (ex >> 16) : (ex & 0xffffu);
                            const int H = ((int)(en << 17)) >> 29;
                            ev[cc][h] = (unsigned)(H * 32) << ((en & 3u) * 8u);
                            ea[cc][h] = en & 0x7fcu;
                        }
                    }
                    asm volatile("s_waitcnt lgkmcnt(0)" ::: "memory");
                    u32x4 o4;
                    o4.x = (p01.x & 0x9f9f9f9fu) + Bv[cc];
                    o4.y = (p01.y & 0x9f9f9f9fu) + Bv[cc];
                    o4.z = (p2 & 0x9f9f9f9fu) + Bv[cc];
                    o4.w = (((p01.x >> 5) & 0x03030303u) | ((p01.y >> 3) & 0x0c0c0c0cu) | ((p2 >> 1) & 0x10101010u) | ((p2 << 1) & 0x80808080u)) + Bv[cc];
                    *(u32x4*)(pb + lane * 16) = o4;
#pragma unroll
                    for (int h = 0; h < 2; ++h)
                        if (ev[cc][h] != 0u && (ea[cc][h] >> 10) == (unsigned)(q & 1))
                            __hip_atomic_fetch_add((unsigned*)(pb + (ea[cc][h] & 0x3fcu)), ev[cc][h], RLX_WG);
                }
                CBAR();
                lds_st(ctl + (L_DEC + ew) * 4, (unsigned)(U * 2 + cc + 9));
            }
        }
        return;
    } else if (wave == 1) {
        const int dest = (b + idx) & 7;
        const int xr = idx & 3, xs = (idx >> 2) & 3, xh = idx >> 4;
        const uint8_t* src = px + (size_t)(dest * 512 + xr * 128 + xh * 64 + (lane & 15)) * 512 + xs * 128 + (lane >> 4) * 16;
        unsigned cprev = __hip_atomic_load(ctr, RLX_AGT);
        u32x4 v[8];
#pragma unroll
        for (int t = 0; t < 4; ++t)
#pragma unroll
            for (int h = 0; h < 2; ++h) v[t * 2 + h] = *(const u32x4*)(src + (size_t)t * 16 * 512 + h * 64);
        WG_BARRIER(cprev);
        call = uni(cprev) + 1u;
        __amdgpu_buffer_rsrc_t rd = __builtin_amdgcn_make_buffer_rsrc(dws + X_OFF + (call & 1u) * XBUF, 0, (unsigned)XBUF, 0x00020000);
        const unsigned obase = (unsigned)rank * 262144u + (unsigned)xs * 65536u + (unsigned)(xr * 8 + xh * 4) * 2048u + lane * 16u;
#pragma unroll
        for (int u = 0; u < 8; ++u) {
            __builtin_amdgcn_raw_buffer_store_b128(v[u], rd, obase + (unsigned)u * 1024u, 0, AUX_SYS);
            if (u > 0) VM_DRAIN();
        }
        if (lane == 0) __hip_atomic_store((unsigned*)(dws + XFLAG_OFF) + xr * 64 + (rank * 4 + xs) * 2 + xh, call, RLX_SYS);
    } else {
        unsigned cprev = __hip_atomic_load(ctr, RLX_AGT);
        WG_BARRIER(cprev);
        call = uni(cprev) + 1u;
    }
    if (lane == 0) __hip_atomic_store(ctr, call, RLX_AGT);
    const unsigned buf = call & 1u;

    if (wave >= 4) {
        const unsigned xcc = __builtin_amdgcn_s_getreg(20 | (3 << 11)) & 15u;
        const u64* invw = (const u64*)(me + INV_OFF) + xcc * 16;
        const u64* wflags = (const u64*)(me + WFLAG_OFF) + buf * 2048 + G * 32;
        const int nsrc = (rank + 1 + (lane + G) % 7) & 7, nkc = (lane / 7) & 7;
        if (wave == 4) {
            const u64 ra = (u64)(p.ws[nsrc] + P_OFF + (size_t)buf * PBUF + (size_t)(G * 8 + nkc) * REC);
            const unsigned alo = (unsigned)ra, ahi = (unsigned)(ra >> 32);
            u64 fv;
            sleep_until(tstart + POLL_START);
            const bool ok = wait_flags64<5>(lane == 8 ? invw : wflags + nsrc * 4 + (nkc >> 1), lane < 8, lane == 8, call, tmo, &fv);
            int issued = 0, pub = 0;
            if (ok) {
                unsigned cd = ((unsigned)(fv >> 32) >> ((nkc & 1) * 6)) & 63u;
                if (lane < 8) lds_st(lds0 + L_CODE + lane * 4, cd);
                lds_st(ctl + L_START * 4, (unsigned)__builtin_amdgcn_s_memrealtime() | 1u);
                if ((unsigned)__builtin_amdgcn_readlane((int)(unsigned)fv, 8) != call) {
                    __builtin_amdgcn_fence(__ATOMIC_ACQUIRE, "");
                    VM_DRAIN();
                }
                unsigned mev = mask_even(cd), mod = mask_odd(cd);
                int tau16 = 0;
                u64 ad[8];
                unsigned me_[8], mo_[8];
#pragma unroll 1
                for (int U = 0; U < 28; ++U) {
                    const int q = U & 3;
                    if (q == 0) {
                        if (U == 4) {
                            wait_lds<false>(ctl + L_TAU * 4, 1u, abortw);
                            if (uni(lds_ld(abortw)) != 0u) break;
                            cd = lds_ld(lds0 + L_CODE + lane * 4);
                            mev = mask_even(cd);
                            mod = mask_odd(cd);
                        }
#pragma unroll
                        for (int c = 0; c < 8; ++c) {
                            const int n = U * 2 + c;
                            ad[c] = (u64)(unsigned)__builtin_amdgcn_readlane((int)alo, n) | ((u64)(unsigned)__builtin_amdgcn_readlane((int)ahi, n) << 32);
                            me_[c] = (unsigned)__builtin_amdgcn_readlane((int)mev, n);
                            mo_[c] = (unsigned)__builtin_amdgcn_readlane((int)mod, n);
                        }
                    }
                    unsigned mk[8];
#pragma unroll
                    for (int c = 0; c < 8; ++c) mk[c] = 0xffff0000u | (((q & 1) ? mo_[c] : me_[c]) >> ((q >> 1) * 16));
                    const unsigned voff = wire_off(q, lane);
                    const int J = 4 + U;
                    const unsigned dst = lds0 + (unsigned)(J % RING) * 8192u;
                    if (J >= RING) wait_lds<true>(ctl + L_CONS * 4, (unsigned)(J - RING + 1), abortw);
                    const unsigned anch = uni(lds_ld(ctl + L_ANCH * 4));
                    if (tau16 == 0) tau16 = (int)uni(lds_ld(ctl + L_TAU * 4));
                    const int lag = (anch && tau16) ? (int)((unsigned)__builtin_amdgcn_s_memrealtime() - anch) - (PACE_D0 + ((U * tau16) >> 4)) : 0;
                    if (lag > PACE_T2) unit8<13>(ad, mk, voff, dst, ctl, issued, pub);
                    else if (lag > PACE_T1) unit8<11>(ad, mk, voff, dst, ctl, issued, pub);
                    else if (lag < -PACE_T2) unit8<7>(ad, mk, voff, dst, ctl, issued, pub);
                    else if (lag < -PACE_T1) unit8<9>(ad, mk, voff, dst, ctl, issued, pub);
                    else unit8<10>(ad, mk, voff, dst, ctl, issued, pub);
                }
            } else {
                lds_st(abortw, 1u);
                lds_st(ctl + L_START * 4, 1u);
            }
            drain<11>(ctl, pub);
        } else if (wave == 10) {
            wait_lds<false>(ctl + L_START * 4, 1u, abortw);
            const unsigned st = lds_ld(ctl + L_START * 4);
            u64 old = 0, fv;
            const u64 key = ((u64)call << 32) | (u64)(0xffffffffu - st);
            if (lane == 0) old = __hip_atomic_fetch_max((u64*)(me + ANCH_OFF), key, RLX_AGT);
            const bool ok = wait_flags64<12>(wflags + nsrc * 4 + (nkc >> 1), lane < 56, false, call, tmo, &fv);
            const unsigned cd = ((unsigned)(fv >> 32) >> ((nkc & 1) * 6)) & 63u;
            if (ok) lds_st(lds0 + L_CODE + lane * 4, cd);
            else lds_st(abortw, 1u);
            const unsigned ca = cd & 7u, cb = cd >> 3;
            const unsigned ln = wave_scan(lane < 56 ? 24u + (ca >= 4u ? 2u : 0u) + (cb >= 4u ? 2u : 0u) + ((ca | cb) >= 4u ? 4u : max(ca, cb) + 1u) : 0u);
            lds_st(ctl + L_TAU * 4, (unsigned)((__builtin_amdgcn_readlane((int)ln, 63) * PACE_TPL) / 1750));
            if (lane == 0) {
                unsigned an = 0xffffffffu - (unsigned)(old > key ? old : key);
                if (st - an > 100000u) an = st;
                lds_st(ctl + L_ANCH * 4, an | 1u);
            }
        } else {
            const unsigned* xf = (const unsigned*)(me + XFLAG_OFF) + rg * 64 + lane;
            if (b < 8) {
                __builtin_amdgcn_fence(__ATOMIC_ACQUIRE, "");
                VM_DRAIN();
                if (lane == 0) __hip_atomic_store((unsigned*)invw, call, RLX_AGT);
            }
            u64 fv;
            sleep_until(tstart + POLL_START + 150);
            bool ok = wait_flags64<20>(lane == 8 ? invw : wflags + rank * 4 + (lane & 3), lane < 4, lane == 8, call, tmo, &fv);
            if (ok) {
                if ((unsigned)__builtin_amdgcn_readlane((int)(unsigned)fv, 8) != call) {
                    __builtin_amdgcn_fence(__ATOMIC_ACQUIRE, "");
                    VM_DRAIN();
                }
                const char* sb = me + W_OFF + (size_t)buf * WBUF + (size_t)G * 32768;
#pragma unroll 1
                for (int q = 0; q < 4; ++q)
#pragma unroll
                    for (int c = 0; c < 8; ++c) glds_piece(lane * 16u, sb + c * 4096 + q * 1024, lds0 + q * 8192 + c * 1024, 0xffffffffu);
                VM_DRAIN();
                ok = wait_flags32<24>(xf, (lane >> 3) == rank, call, tmo);
            }
            if (!ok) lds_st(abortw, 1u);
            lds_st(ctl + L_XOWN * 4, 1u);
            if (ok) ok = wait_flags32<24>(xf, true, call, tmo);
            if (!ok) lds_st(abortw, 1u);
            lds_st(ctl + L_XALL * 4, 1u);
        }
        return;
    }

    const float scale = p.sx[0] * p.sw[0];
    f32x4 acc[2][4];
#pragma unroll
    for (int t = 0; t < 2; ++t)
#pragma unroll
        for (int q = 0; q < 4; ++q) acc[t][q] = (f32x4){0.f, 0.f, 0.f, 0.f};
    const char* xb = me + X_OFF + (size_t)buf * XBUF + (size_t)(rg * 8 + wave * 2) * 2048 + lane * 16;
    i32x4 A[8][2];
    int pi = G % 7, pk = 0, pc = 0;
    auto loadA1 = [&](int c) {
        const char* a = xb + (size_t)((rank + 1 + pi) & 7) * 262144 + (size_t)((pk >> 1) & 3) * 65536 + (pk & 1) * 1024;
        A[c][0] = *(const i32x4*)a;
        A[c][1] = *(const i32x4*)(a + 2048);
        if (++pi == 7) pi = 0;
        if (++pc == 7) { pc = 0; ++pk; }
    };
    const unsigned wofs = (unsigned)(((lane & 3) * 16) + (((lane >> 4) ^ (lane & 3)) * 4) + ((lane >> 2) & 3)) * 16u;
    __amdgpu_buffer_rsrc_t ro = __builtin_amdgcn_make_buffer_rsrc(p.out, 0, 512u * 4096u * 4u, 0x00020000);
    const unsigned ob = ((unsigned)(rg * 128 + wave * 32 + (lane & 15)) * 4096u + G * 64 + (lane >> 4) * 4) * 4u;
    unsigned seen = 8u;
    auto sstep = [&](int J0, int MODE) {
#pragma unroll
        for (int q = 0; q < 4; ++q) {
            const int J = J0 + q;
            const char* bs = lds + (J % RING) * 8192 + wofs;
            unsigned ab = 0u;
#pragma unroll
            for (int hh = 0; hh < 2; ++hh) {
                if (MODE != 0) {
                    if (seen < (unsigned)(J * 2 + hh + 1)) seen = wait_lds<true>(ctl + L_DEC * 4, (unsigned)(J * 2 + hh + 1), abortw);
                    CBAR();
                }
                if (MODE == 2 && hh == 1) ab = *(const unsigned*)(lds + L_CTL + L_ABORT * 4);
                i32x4 wf[4];
#pragma unroll
                for (int c = 0; c < 4; ++c) wf[c] = *(const i32x4*)(bs + (hh * 4 + c) * 1024);
#pragma unroll
                for (int c = 0; c < 4; c += 2) {
                    const i32x8 W = __builtin_shufflevector(wf[c], wf[c + 1], 0, 1, 2, 3, 4, 5, 6, 7);
#pragma unroll
                    for (int t = 0; t < 2; ++t) {
                        const i32x8 a = __builtin_shufflevector(A[hh * 4 + c][t], A[hh * 4 + c + 1][t], 0, 1, 2, 3, 4, 5, 6, 7);
                        acc[t][q] = __builtin_amdgcn_mfma_scale_f32_16x16x128_f8f6f4(W, a, acc[t][q], 1, 1, 0, 0x7f7f7f7f, 0, 0x7f7f7f7f);
                    }
                }
                asm volatile("" : "+v"(acc[0][q]), "+v"(acc[1][q]) :: "memory");
                if (MODE == 1 && q == 3) {
#pragma unroll
                    for (int c = hh * 4; c < hh * 4 + 4; ++c) loadA1(c);
                }
            }
            if (MODE != 2) {
                lds_st(ctl + (L_CONS + wave) * 4, (unsigned)(J + 1));
            } else {
#pragma unroll
                for (int t = 0; t < 2; ++t) {
                    f32x4 y;
#pragma unroll
                    for (int r = 0; r < 4; ++r) {
                        const float v = acc[t][q][r] * scale;
                        const float cl = fminf(fmaxf(v, -60.f), 60.f);
                        y[r] = ab ? __builtin_nanf("") : v * __builtin_amdgcn_rcpf(1.f + __expf(-cl));
                    }
                    __builtin_amdgcn_raw_buffer_store_b128(__builtin_bit_cast(u32x4, y), ro, ob + (unsigned)t * 16u * 16384u + (unsigned)q * 64u, 0, AUX_AGT);
                }
                CBAR();
            }
        }
    };
    wait_lds<false>(ctl + L_XOWN * 4, 1u, abortw);
    CBAR();
#pragma unroll
    for (int c = 0; c < 8; ++c) {
        const char* a = xb + (size_t)rank * 262144 + (size_t)(c >> 1) * 65536 + (c & 1) * 1024;
        A[c][0] = *(const i32x4*)a;
        A[c][1] = *(const i32x4*)(a + 2048);
    }
    sstep(0, 0);
    wait_lds<false>(ctl + L_XALL * 4, 1u, abortw);
    CBAR();
#pragma unroll
    for (int c = 0; c < 8; ++c) loadA1(c);
#pragma unroll 1
    for (int J0 = 4; J0 < 28; J0 += 4) sstep(J0, 1);
    sstep(28, 2);
}

extern "C" void kernel_launch(int rank, int nranks, ncclComm_t comm, hipStream_t stream,
                              void* const* d_in, const int* in_sizes, int n_in,
                              void* d_out, int out_size,
                              const int* in_dtypes, int out_dtype,
                              void* const* d_out_all,
                              void* const* d_ws_all, size_t ws_bytes) {
    if (nranks != NR || (unsigned)rank >= (unsigned)NR || ws_bytes < WS_NEED || n_in < 4 ||
        in_sizes[0] != 4096 * 512 || in_sizes[1] != 512 * 4096 || out_size != 512 * 4096) {
        fprintf(stderr, "kernel_launch: unexpected problem geometry\n");
        return;
    }
    Params p;
    p.x = (const uint8_t*)d_in[0];
    p.w = (const uint8_t*)d_in[1];
    p.sx = (const float*)d_in[2];
    p.sw = (const float*)d_in[3];
    p.out = (float*)d_out;
    for (int r = 0; r < NR; ++r) p.ws[r] = (char*)d_ws_all[r];
    p.me = p.ws[rank];
    p.rank = rank;
    p.pad = 0;
    p.pad2 = nullptr;
    hipLaunchKernelGGL(fused_gemm_silu, dim3(NWG), dim3(NTHREADS), 0, stream, p);
}
```
